```python
import numpy as np

kernel_name = "gpu_gemm_a2a_m4096_k4096_n4096_bf16_reluqe4m3"


ABSMAX_RTOL = 0.09

COMPUTE_DTYPE = "bfloat16"
EPILOGUE = "relu_quant_e4m3"
IN_QUANT = None
OUT_QUANT = "e4m3"
IN_DTYPES = {"x": "bfloat16", "w_mat": "bfloat16"}
WORLD_SIZE = 8

def _to_bf16(x):
    u = np.ascontiguousarray(x, dtype=np.float32).view(np.uint32)
    return ((u + 0x7FFF + ((u >> 16) & 1)) & 0xFFFF0000).view(np.float32)

def setup_inputs():

    x = _to_bf16(np.random.randn(4096, 4096).astype(np.float32))
    w_mat = _to_bf16((np.random.randn(4096, 4096) / np.sqrt(4096)).astype(np.float32))
    return {"x": x, "w_mat": w_mat}

def _snap_fp8(x, mant_bits, max_abs, min_normal):
    a = np.abs(x.astype(np.float64))
    _m, _e = np.frexp(a)
    q = np.where(a >= min_normal, np.ldexp(1.0, _e - 1 - mant_bits),
                 min_normal * 2.0 ** -mant_bits)
    snapped = np.minimum(np.round(a / q) * q, max_abs)
    return (np.sign(x) * snapped).astype(np.float32)
def _quant_dequant(y, amax):
    scale = amax / 448.0
    q = _snap_fp8(y / scale, 3, 448.0, 2.0 ** -6)
    return (q * scale).astype(np.float32)
def _epilogue(y):
    y = np.maximum(y, 0.0)
    return _quant_dequant(y, np.abs(y).max())

def reference(x, w_mat):
    y = (x @ w_mat).astype(np.float32)
    return _epilogue(y)
```

```cpp
#include <hip/hip_runtime.h>
#include <rccl/rccl.h>
#include <cstdio>

#define HIP_CHECK(expr) do { hipError_t _e = (expr); if (_e != hipSuccess) { \
    fprintf(stderr, "HIP error %s at %s:%d\n", hipGetErrorString(_e), __FILE__, __LINE__); \
    return; } } while (0)

#define NP2_STEPS 8

namespace ga {

#define LAS __attribute__((address_space(3)))
#define GAS __attribute__((address_space(1)))
typedef __bf16 bf16x4 __attribute__((ext_vector_type(4)));
typedef __bf16 bf16x8 __attribute__((ext_vector_type(8)));
typedef float f32x4 __attribute__((ext_vector_type(4)));
typedef float f32x16 __attribute__((ext_vector_type(16)));
typedef unsigned u32x4 __attribute__((ext_vector_type(4)));
typedef unsigned long long u64;

constexpr int NR = 8;
constexpr int ML = 512, NN = 4096, KK = 4096, NL = 512;
constexpr int BK = 64, NKS = KK / BK;
constexpr int NP2 = NP2_STEPS;
constexpr int NP1 = (NKS - NP2) / 2;
constexpr int NS = NP1 + NP2;
static_assert(2 * NP1 + NP2 == NKS && NP2 >= 6, "K split");
constexpr int ST = 4, STAGE_B = 32768, W_OFF = 16384;
constexpr int R2_OFF = ST * STAGE_B;
constexpr int LDS_BYTES = R2_OFF + 32768;
constexpr int GRID = 256, THREADS = 512;
constexpr int FLAG_SLOT = 2, POLL_SLOT = 3;

constexpr size_t WS_CNT = 0x0, WS_TMO = 0x1000, WS_LMAX = 0x2000, WS_GMAX = 0x4000, WS_DONE = 0x6000,
                 WS_XFLAG = 0x8000, WS_STAMP = 0x10000, WS_XCH = 0x100000, WS_END = WS_XCH + (size_t)GRID * 32768;
constexpr int NREP = 8;
constexpr u64 DEADLINE_TICKS = 50000000ull;

struct Params {
    const void* x;
    const void* w;
    unsigned char* out[NR];
    unsigned char* ws[NR];
    int rank;
    int pad;
};

#define RLX_AGENT __ATOMIC_RELAXED, __HIP_MEMORY_SCOPE_AGENT
#define RLX_SYS __ATOMIC_RELAXED, __HIP_MEMORY_SCOPE_SYSTEM
#define BARRIER() asm volatile("s_barrier" ::: "memory")
#define WAIT_VM(n) asm volatile("s_waitcnt vmcnt(" #n ")" ::: "memory")
#define WAIT_LGKM0() asm volatile("s_waitcnt lgkmcnt(0)" ::: "memory")

#define STAMP(i) do {} while (0)

#if defined(__HIP_DEVICE_COMPILE__)
__device__ __forceinline__ bool give_up(GAS unsigned* tmo, unsigned call, u64 t0) {
    if (__hip_atomic_load(tmo, RLX_AGENT) == call || __builtin_amdgcn_s_memrealtime() - t0 > DEADLINE_TICKS) {
        __hip_atomic_store(tmo, call, RLX_AGENT);
        return true;
    }
    __builtin_amdgcn_s_sleep(1);
    return false;
}
constexpr int AUX_SYS = 17;
__device__ __forceinline__ unsigned wave_umax(unsigned v) {
#pragma unroll
    for (int o = 32; o > 0; o >>= 1) { unsigned t = __shfl_xor(v, o, 64); v = v > t ? v : t; }
    return v;
}
#endif

__global__ __launch_bounds__(THREADS, 2) void gemm_a2a_kernel(Params p) {
#if defined(__HIP_DEVICE_COMPILE__)
    extern __shared__ __attribute__((aligned(16))) char shm_raw[];
    LAS char* shm = (LAS char*)shm_raw;
    const int b = blockIdx.x;
    const int grp = b & 7, idx = b >> 3;
    const int tn = idx & 3, tm = (idx >> 2) & 3, hb = idx >> 4;
    const int wid = __builtin_amdgcn_readfirstlane(threadIdx.x >> 6);
    const int lane = threadIdx.x & 63;
    GAS unsigned char* myws = (GAS unsigned char*)p.ws[p.rank & 7];
    GAS unsigned* tmo = (GAS unsigned*)(myws + WS_TMO);
    GAS unsigned* cnt = (GAS unsigned*)(myws + WS_CNT) + b;
    STAMP(0);

    constexpr int R1_OFF = (NS % ST) * STAGE_B;
    constexpr int R3_OFF = ((NS + 1) % ST) * STAGE_B;
    constexpr int SM_OFF = ((NS + 2) % ST) * STAGE_B;

    unsigned call;
    f32x16 acc[2][2];
    const int wr = (wid >> 1) & 1, wc = wid & 1;

    if (wid >= 4 && wid < 6) {
        const int lw = wid - 4;
        auto rx = __builtin_amdgcn_make_buffer_rsrc((void*)p.x, 0, ML * KK * 2, 0x00020000);
        auto rw = __builtin_amdgcn_make_buffer_rsrc((void*)p.w, 0, KK * NN * 2, 0x00020000);
        const int vx = (tm * 128 + lw * 8 + (lane >> 3)) * (KK * 2) + (((lane & 7) ^ ((lw << 2) | (lane >> 4))) << 4);
        const int wrow = (lw * 4 + (lane >> 4)) * (NN * 2) + (grp * NL + tn * 128) * 2;
        const int vw0 = wrow + (((lane & 15) ^ (((lane >> 4) << 2) | lw)) << 4);
        const int vw1 = wrow + (((lane & 15) ^ (((lane >> 4) << 2) | (2 + lw))) << 4);
        int nl, no, q, cm, lead2, nt, kb;
        auto order_init = [&](int q_, int ncls, int lead, int nt_, int kb_) {
            q = q_; cm = ncls - 1; lead2 = lead; nt = nt_; kb = kb_; nl = q_; no = (q_ == 0) ? 1 : 0;
        };
        auto order_next = [&]() {
            int t;
            if (nl < nt && (nl - lead2 <= no || no >= nt)) { t = nl; nl += cm + 1; }
            else { t = no; ++no; if ((no & cm) == q) ++no; }
            return kb + t;
        };
        auto issue = [&](int s) {
            if (s == 0) order_init((tm + tn) & 3, 4, 3, NP1, hb * NP1);
            if (s == NP1) order_init((tm * 2 + hb + 2 * tn) & 7, 8, 4, NP2, 2 * NP1);
            const int t = order_next();
            LAS char* dst = shm + (s & (ST - 1)) * STAGE_B + lw * 1024;
            const int sx = t * (BK * 2), sw = t * (BK * NN * 2);
            if (s < NP1) {
#pragma unroll
                for (int i = 0; i < 8; ++i)
                    __builtin_amdgcn_raw_ptr_buffer_load_lds(rx, dst + i * 2048, 16, vx, sx + i * (16 * KK * 2), 0, 0);
            } else {
#pragma unroll
                for (int i = 0; i < 4; ++i)
                    __builtin_amdgcn_raw_ptr_buffer_load_lds(rx, dst + i * 2048, 16, vx, sx + (hb * 64 + i * 16) * (KK * 2), 0, 0);
            }
#pragma unroll
            for (int i = 0; i < 8; ++i)
                __builtin_amdgcn_raw_ptr_buffer_load_lds(rw, dst + W_OFF + i * 2048, 16, (i & 1) ? vw1 : vw0, sw + i * (8 * NN * 2), 0, 0);
        };
        issue(0); issue(1); issue(2);
        unsigned c0 = __hip_atomic_load(cnt, RLX_AGENT);
        WAIT_VM(24);
        for (int s = 0; s < NS - 3; ++s) {
            BARRIER();
            issue(s + 3);
            WAIT_VM(24);
        }
        BARRIER(); WAIT_VM(12);
        BARRIER(); WAIT_VM(0);
        BARRIER();
        call = __builtin_amdgcn_readfirstlane(c0) + 1;
    } else if (wid >= 6) {
        call = __builtin_amdgcn_readfirstlane(__hip_atomic_load(cnt, RLX_AGENT)) + 1;
        if (wid == 6) {
            GAS u64* xf = (GAS u64*)(myws + WS_XFLAG) + (b ^ 128);
            auto rp = __builtin_amdgcn_make_buffer_rsrc((void*)(p.ws[p.rank & 7] + WS_XCH + (size_t)(b ^ 128) * 32768), 0, 32768, 0x00020000);
            auto fetch = [&]() {
#pragma unroll
                for (int i = 0; i < 32; ++i)
                    __builtin_amdgcn_raw_ptr_buffer_load_lds(rp, shm + R2_OFF + i * 1024, 16, lane * 16, i * 1024, 0, AUX_SYS);
            };
            auto both = [&](u64 v) { return __builtin_amdgcn_readfirstlane((int)((unsigned)v >= call && (unsigned)(v >> 32) >= call)) != 0; };
            bool got = false;
            for (int s = 0; s <= NP1 + POLL_SLOT; ++s) BARRIER();
            constexpr int NPOLL = (NP2 - 1 - POLL_SLOT) / 2;
#pragma unroll
            for (int k = 0; k < NPOLL; ++k) {
                if (!got) {
                    u64 f;
                    asm volatile("global_load_dwordx2 %0, %1, off sc0 sc1\n\ts_barrier\n\ts_barrier\n\ts_waitcnt vmcnt(0)"
                                 : "=&v"(f) : "v"(xf) : "memory");
                    if (both(f)) { fetch(); got = true; }
                } else {
                    BARRIER(); BARRIER();
                }
            }
            for (int s = NP1 + POLL_SLOT + 1 + 2 * NPOLL; s < NS; ++s) BARRIER();
            if (!got) {
                const u64 t0 = __builtin_amdgcn_s_memrealtime();
                bool ok = true;
                while (!both(__hip_atomic_load(xf, RLX_SYS)))
                    if (__builtin_amdgcn_readfirstlane((int)give_up(tmo, call, t0))) { ok = false; break; }
                if (ok) fetch();
            }
            WAIT_VM(0);
        } else {
            for (int s = 0; s < NS; ++s) BARRIER();
        }
    } else {
        unsigned c0 = __hip_atomic_load(cnt, RLX_AGENT);
        const int h = lane >> 5, blk = (lane >> 4) & 1, qq = (lane & 15) >> 2, pp = lane & 3;
        int xo[4], wo[2][2];
#pragma unroll
        for (int ks = 0; ks < 4; ++ks) xo[ks] = (lane & 31) * 128 + (((2 * ks + h) ^ ((lane >> 1) & 7)) << 4);
#pragma unroll
        for (int j = 0; j < 2; ++j)
#pragma unroll
            for (int t = 0; t < 2; ++t)
                wo[j][t] = W_OFF + 256 * (8 * h + 4 * t + qq) +
                           (((4 * (wc * 2 + j) + 2 * blk + (pp >> 1)) ^ ((qq << 2) | ((2 * h + t) & 3))) << 4) + 8 * (pp & 1);
#pragma unroll
        for (int i = 0; i < 2; ++i)
#pragma unroll
            for (int j = 0; j < 2; ++j)
#pragma unroll
                for (int r = 0; r < 16; ++r) acc[i][j][r] = 0.f;

        struct Frag { bf16x8 x[2], w[2]; };
        auto rd = [&](Frag& f, LAS char* xb, LAS char* wb) {
            f.x[0] = *(LAS bf16x8*)(xb);
            f.x[1] = *(LAS bf16x8*)(xb + 4096);
#pragma unroll
            for (int j = 0; j < 2; ++j) {
                const bf16x4 lo = __builtin_amdgcn_ds_read_tr16_b64_v4bf16((LAS bf16x4*)(wb + wo[j][0]));
                const bf16x4 hi = __builtin_amdgcn_ds_read_tr16_b64_v4bf16((LAS bf16x4*)(wb + wo[j][1]));
                f.w[j] = __builtin_shufflevector(lo, hi, 0, 1, 2, 3, 4, 5, 6, 7);
            }
        };
        auto mm = [&](const Frag& f) {
#pragma unroll
            for (int i = 0; i < 2; ++i)
#pragma unroll
                for (int j = 0; j < 2; ++j)
                    acc[i][j] = __builtin_amdgcn_mfma_f32_32x32x16_bf16(f.w[j], f.x[i], acc[i][j], 0, 0, 0);
        };
#define GROUP_END() do { \
            __builtin_amdgcn_sched_group_barrier(0x8, 1, 0); __builtin_amdgcn_sched_group_barrier(0x100, 2, 0); \
            __builtin_amdgcn_sched_group_barrier(0x8, 1, 0); __builtin_amdgcn_sched_group_barrier(0x100, 2, 0); \
            __builtin_amdgcn_sched_group_barrier(0x8, 1, 0); __builtin_amdgcn_sched_group_barrier(0x100, 2, 0); \
            __builtin_amdgcn_sched_group_barrier(0x8, 1, 0); __builtin_amdgcn_sched_barrier(0); } while (0)
        Frag fa, fb, fc, fd;

        {
            BARRIER();
            LAS char* xb = shm + wr * 8192;
            rd(fa, xb + xo[0], shm);
            rd(fb, xb + xo[1], shm + 4096);
            __builtin_amdgcn_sched_barrier(0);
            rd(fc, xb + xo[2], shm + 8192); mm(fa); GROUP_END();
            rd(fd, xb + xo[3], shm + 12288); mm(fb); GROUP_END();
            WAIT_LGKM0();
        }
        for (int s = 1; s < NP1; ++s) {
            BARRIER();
            __builtin_amdgcn_sched_barrier(0);
            LAS char* sb = shm + (s & (ST - 1)) * STAGE_B;
            LAS char* xb = sb + wr * 8192;
            rd(fa, xb + xo[0], sb); mm(fc); GROUP_END();
            rd(fb, xb + xo[1], sb + 4096); mm(fd); GROUP_END();
            rd(fc, xb + xo[2], sb + 8192); mm(fa); GROUP_END();
            rd(fd, xb + xo[3], sb + 12288); mm(fb); GROUP_END();
            WAIT_LGKM0();
        }
        mm(fc); mm(fd);
        STAMP(1);
        call = __builtin_amdgcn_readfirstlane(c0) + 1;
        const bool giver = (wr != hb);
        if (giver) {
            auto rdump = __builtin_amdgcn_make_buffer_rsrc((void*)(p.ws[p.rank & 7] + WS_XCH + (size_t)b * 32768), 0, 32768, 0x00020000);
#pragma unroll
            for (int i = 0; i < 2; ++i)
#pragma unroll
                for (int j = 0; j < 2; ++j)
#pragma unroll
                    for (int g = 0; g < 4; ++g) {
                        f32x4 v = {acc[i][j][4 * g], acc[i][j][4 * g + 1], acc[i][j][4 * g + 2], acc[i][j][4 * g + 3]};
                        __builtin_amdgcn_raw_buffer_store_b128(__builtin_bit_cast(u32x4, v), rdump, lane * 16, wc * 16384 + ((i * 2 + j) * 4 + g) * 1024, AUX_SYS);
                    }
            asm volatile("s_nop 1" ::: "memory");
            __builtin_amdgcn_sched_barrier(0);
#pragma unroll
            for (int i = 0; i < 2; ++i)
#pragma unroll
                for (int j = 0; j < 2; ++j)
#pragma unroll
                    for (int r = 0; r < 16; ++r) acc[i][j][r] = 0.f;
        }
        const int x2a = wr ? xo[2] : xo[0], x2b = wr ? xo[3] : xo[1];
        static_assert(NP2 % 2 == 0, "phase 2 is unrolled in pairs");
#pragma unroll
        for (int u = 0; u < NP2; ++u) {
            BARRIER();
            __builtin_amdgcn_sched_barrier(0);
            LAS char* sb = shm + ((NP1 + u) & (ST - 1)) * STAGE_B;
            LAS char* sw = sb + wr * 8192;
            if (u == 0) {
                rd(fa, sb + x2a, sw); rd(fb, sb + x2b, sw + 4096);
            } else if (u & 1) {
                rd(fc, sb + x2a, sw); mm(fa); GROUP_END();
                rd(fd, sb + x2b, sw + 4096); mm(fb); GROUP_END();
            } else {
                rd(fa, sb + x2a, sw); mm(fc); GROUP_END();
                rd(fb, sb + x2b, sw + 4096); mm(fd); GROUP_END();
            }
            WAIT_LGKM0();
            if (u == FLAG_SLOT && giver) {
                WAIT_VM(0);
                if (lane == 0) __hip_atomic_store((GAS unsigned*)(myws + WS_XFLAG) + b * 2 + wc, call, RLX_SYS);
            }
        }
        mm(fc); mm(fd);
        STAMP(2);
#pragma unroll
        for (int j = 0; j < 2; ++j)
#pragma unroll
            for (int g = 0; g < 4; ++g) {
                f32x4 v;
#pragma unroll
                for (int e = 0; e < 4; ++e) v[e] = wr ? acc[0][j][4 * g + e] : acc[1][j][4 * g + e];
                *(LAS f32x4*)(shm + R1_OFF + (((wc * 2 + (1 - wr)) * 2 + j) * 4 + g) * 1024 + lane * 16) = v;
            }
        WAIT_LGKM0();
    }
    BARRIER();
    if (wid < 4) {
        unsigned mx = 0;
#pragma unroll
        for (int j = 0; j < 2; ++j)
#pragma unroll
            for (int g = 0; g < 4; ++g) {
                const int blkoff = (((wc * 2 + wr) * 2 + j) * 4 + g) * 1024;
                const f32x4 a = *(LAS f32x4*)(shm + R1_OFF + blkoff + lane * 16);
                const f32x4 c = *(LAS f32x4*)(shm + R2_OFF + blkoff + lane * 16);
                f32x4 v;
#pragma unroll
                for (int e = 0; e < 4; ++e) {
                    const float own = wr ? acc[1][j][4 * g + e] : acc[0][j][4 * g + e];
                    v[e] = fmaxf(own + a[e] + c[e], 0.f);
                    const unsigned u = __float_as_uint(v[e]) & 0x7fffffffu;
                    mx = mx > u ? mx : u;
                }
                *(LAS f32x4*)(shm + R3_OFF + blkoff + ((lane ^ ((j * 2 + (g >> 1)) << 2)) << 4)) = v;
            }
        mx = wave_umax(mx);
        if (lane == 0) *(LAS unsigned*)(shm + SM_OFF + wid * 4) = mx;
        WAIT_LGKM0();
    }
    BARRIER();
    STAMP(3);
    if (wid == 7) {
        const u32x4 m4 = *(LAS u32x4*)(shm + SM_OFF);
        unsigned mx = m4[0] > m4[1] ? m4[0] : m4[1];
        mx = mx > m4[2] ? mx : m4[2];
        mx = mx > m4[3] ? mx : m4[3];
        GAS u64* lmax = (GAS u64*)(myws + WS_LMAX);
        if (lane == 0) __hip_atomic_store(lmax + b, ((u64)call << 32) | mx, RLX_AGENT);
        const u64 t0 = __builtin_amdgcn_s_memrealtime();
        const size_t goff = WS_GMAX + (size_t)(call & 1) * (NREP * 128);
        if (b == 0) {
            unsigned gm;
            for (;;) {
                bool ok = true;
                gm = 0;
#pragma unroll
                for (int k = 0; k < 4; ++k) {
                    const u64 v = __hip_atomic_load(lmax + k * 64 + lane, RLX_AGENT);
                    ok &= (unsigned)(v >> 32) >= call;
                    gm = gm > (unsigned)v ? gm : (unsigned)v;
                }
                if (__all(ok)) break;
                if (__builtin_amdgcn_readfirstlane((int)give_up(tmo, call, t0))) break;
            }
            gm = wave_umax(gm);
            unsigned char* pw = p.ws[0];
#pragma unroll
            for (int r = 1; r < NR; ++r) pw = (lane >> 3) == r ? p.ws[r] : pw;
            if (__hip_atomic_load(tmo, RLX_AGENT) != call)
                __hip_atomic_store((GAS u64*)(pw + goff + (lane & 7) * 128) + (p.rank & 7), ((u64)call << 32) | gm, RLX_SYS);
            STAMP(4);
        }
        GAS u64* gp = (GAS u64*)(myws + goff + grp * 128) + (lane & 7);
        unsigned am;
        for (;;) {
            const u64 v = __hip_atomic_load(gp, RLX_SYS);
            am = (unsigned)v;
            if (__all((unsigned)(v >> 32) >= call)) break;
            if (__builtin_amdgcn_readfirstlane((int)give_up(tmo, call, t0))) break;
        }
#pragma unroll
        for (int o = 4; o > 0; o >>= 1) { unsigned t = __shfl_xor(am, o, 64); am = am > t ? am : t; }
        const unsigned bad = __hip_atomic_load(tmo, RLX_AGENT) == call;
        if (lane == 0) { *(LAS unsigned*)(shm + SM_OFF + 16) = am; *(LAS unsigned*)(shm + SM_OFF + 20) = bad; }
        WAIT_LGKM0();
    }
    f32x4 y[4];
    {
        const int m = wid * 8 + (lane >> 3), cg = lane & 7;
        const int base = R3_OFF + ((((cg >> 2) * 2 + (m >> 5)) * 2 + ((cg >> 1) & 1)) * 4 + 2 * (cg & 1)) * 1024;
        const int sx = (cg & 3) << 2;
#pragma unroll
        for (int i = 0; i < 4; ++i)
            y[i] = *(LAS f32x4*)(shm + base + (i >> 1) * 1024 + ((((m & 31) + 32 * (i & 1)) ^ sx) << 4));
    }
    BARRIER();
    STAMP(5);
    {
        const float amax = __uint_as_float(*(LAS unsigned*)(shm + SM_OFF + 16));
        const unsigned bad = *(LAS unsigned*)(shm + SM_OFF + 20);
        const float sc = amax / 448.0f;
        const float inv = sc > 0.f ? 1.0f / sc : 0.f;
        u32x4 o;
#pragma unroll
        for (int i = 0; i < 4; ++i) {
            int w = __builtin_amdgcn_cvt_pk_fp8_f32(y[i][0] * inv, y[i][1] * inv, 0, false);
            w = __builtin_amdgcn_cvt_pk_fp8_f32(y[i][2] * inv, y[i][3] * inv, w, true);
            o[i] = bad ? 0x7f7f7f7fu : (unsigned)w;
        }
        const int row = (p.rank & 7) * ML + tm * 128 + hb * 64 + wid * 8 + (lane >> 3);
        auto ro = __builtin_amdgcn_make_buffer_rsrc((void*)p.out[grp], 0, NR * ML * NL, 0x00020000);
        __builtin_amdgcn_raw_buffer_store_b128(o, ro, row * NL + tn * 128 + (lane & 7) * 16, 0, AUX_SYS);
    }
    WAIT_VM(0);
    BARRIER();
    STAMP(6);
    if (wid == 7) {
        if (lane == 0) {
            __hip_atomic_store((GAS unsigned*)((GAS unsigned char*)p.ws[grp] + WS_DONE) + (p.rank & 7) * 32 + idx, call, RLX_SYS);
            __hip_atomic_store(cnt, call, RLX_AGENT);
        }
        if (b == 0) {
            GAS u64* dn = (GAS u64*)(myws + WS_DONE);
            const u64 t0 = __builtin_amdgcn_s_memrealtime();
            for (;;) {
                const u64 v0 = __hip_atomic_load(dn + lane, RLX_SYS), v1 = __hip_atomic_load(dn + 64 + lane, RLX_SYS);
                const bool ok = (unsigned)v0 >= call && (unsigned)(v0 >> 32) >= call && (unsigned)v1 >= call && (unsigned)(v1 >> 32) >= call;
                if (__all(ok)) break;
                if (__builtin_amdgcn_readfirstlane((int)give_up(tmo, call, t0))) {
                    if (lane == 0)
                        __hip_atomic_store((GAS unsigned*)((GAS unsigned char*)p.out[p.rank & 7] + (size_t)(p.rank & 7) * ML * NL), 0x7f7f7f7fu, RLX_SYS);
                    break;
                }
            }
            STAMP(7);
        }
    }
#endif
}

}

extern "C" void kernel_launch(int rank, int nranks, ncclComm_t comm, hipStream_t stream,
                              void* const* d_in, const int* in_sizes, int n_in,
                              void* d_out, int out_size, const int* in_dtypes, int out_dtype,
                              void* const* d_out_all, void* const* d_ws_all, size_t ws_bytes) {
    (void)comm; (void)in_dtypes; (void)out_dtype; (void)d_out;
    if (nranks != ga::NR || (unsigned)rank >= (unsigned)ga::NR || n_in < 2 || in_sizes[0] != ga::ML * ga::KK ||
        in_sizes[1] != ga::KK * ga::NN || out_size != ga::NR * ga::ML * ga::NL || ws_bytes < ga::WS_END) {
        fprintf(stderr, "kernel_launch: unexpected problem shape\n");
        return;
    }
    HIP_CHECK(hipFuncSetAttribute((const void*)ga::gemm_a2a_kernel, hipFuncAttributeMaxDynamicSharedMemorySize, ga::LDS_BYTES));
    ga::Params p;
    p.x = d_in[0];
    p.w = d_in[1];
    for (int r = 0; r < ga::NR; ++r) {
        p.out[r] = (unsigned char*)d_out_all[r];
        p.ws[r] = (unsigned char*)d_ws_all[r];
    }
    p.rank = rank;
    p.pad = 0;
    hipLaunchKernelGGL(ga::gemm_a2a_kernel, dim3(ga::GRID), dim3(ga::THREADS), ga::LDS_BYTES, stream, p);
    HIP_CHECK(hipGetLastError());
}
```

```cpp
#include <hip/hip_runtime.h>
#include <rccl/rccl.h>
#include <cstdio>

#define HIP_CHECK(expr) do { hipError_t _e = (expr); if (_e != hipSuccess) { \
    fprintf(stderr, "HIP error %s at %s:%d\n", hipGetErrorString(_e), __FILE__, __LINE__); \
    return; } } while (0)

#define NP2_STEPS 12
#define FLAG_SLOT_ 6
#define POLL_SLOT_ 8
#define DUMP_PER_STEP 4
#define LOADER_PRIO 0
#define LEAD1 3
#define LEAD2 4
#define AMAX_FLAT 0
#define MF16 1
#define DIST_ 3
#define LOAD_AUX 0

namespace ga {

#define LAS __attribute__((address_space(3)))
#define GAS __attribute__((address_space(1)))
typedef __bf16 bf16x4 __attribute__((ext_vector_type(4)));
typedef __bf16 bf16x8 __attribute__((ext_vector_type(8)));
typedef float f32x4 __attribute__((ext_vector_type(4)));
typedef float f32x16 __attribute__((ext_vector_type(16)));
typedef unsigned u32x4 __attribute__((ext_vector_type(4)));
typedef unsigned long long u64;

constexpr int NR = 8;
constexpr int ML = 512, NN = 4096, KK = 4096, NL = 512;
constexpr int BK = 64, NKS = KK / BK;
constexpr int NP2 = NP2_STEPS;
constexpr int NP1 = (NKS - NP2) / 2;
constexpr int NS = NP1 + NP2;
static_assert(2 * NP1 + NP2 == NKS && NP2 >= 6, "K split");
constexpr int ST = 4, STAGE_B = 32768, W_OFF = 16384;
constexpr int KB_OFF = ST * STAGE_B;
constexpr int FL_OFF = KB_OFF + 16384;
constexpr int LDS_BYTES = FL_OFF + 256;
constexpr int F_WMAX = 8, F_AMAX = 16, F_BAD = 17, F_CALL = 18, F_R2 = 19;
constexpr int GRID = 256, THREADS = 512, DIST = DIST_;
static_assert(DIST == 2 || DIST == 3, "prefetch distance");
constexpr int FLAG_SLOT = FLAG_SLOT_, POLL_SLOT = POLL_SLOT_;
static_assert(FLAG_SLOT * DUMP_PER_STEP >= 16 && FLAG_SLOT < NP2 && POLL_SLOT + 2 <= NP2, "hand-off schedule");

constexpr size_t WS_CNT = 0x0, WS_TMO = 0x1000, WS_LMAX = 0x2000, WS_GMAX = 0x4000, WS_DONE = 0x6000,
                 WS_XFLAG = 0x8000, WS_STAMP = 0x10000, WS_FLAT = 0xF0000, WS_XCH = 0x100000,
                 WS_END = WS_XCH + (size_t)GRID * 32768;
constexpr int NREP = 8;
constexpr u64 DEADLINE_TICKS = 50000000ull;

struct Params {
    const void* x;
    const void* w;
    unsigned char* out[NR];
    unsigned char* ws[NR];
    int rank;
    int pad;
};

#define RLX_AGENT __ATOMIC_RELAXED, __HIP_MEMORY_SCOPE_AGENT
#define RLX_SYS __ATOMIC_RELAXED, __HIP_MEMORY_SCOPE_SYSTEM
#define BARRIER() asm volatile("s_barrier" ::: "memory")
#define WAIT_VM(n) asm volatile("s_waitcnt vmcnt(" #n ")" ::: "memory")
#define WAIT_LGKM0() asm volatile("s_waitcnt lgkmcnt(0)" ::: "memory")
#define RFL(x) __builtin_amdgcn_readfirstlane(x)

#define SRC_SLOT(d) (p.rank & 7)
#define STAMP(i) do {} while (0)

#if defined(__HIP_DEVICE_COMPILE__)
constexpr int AUX_SYS = 17;
__device__ __forceinline__ bool give_up(GAS unsigned* tmo, unsigned call, u64 t0, unsigned& polls) {
    if ((++polls & 31u) != 0u) return false;
    if (RFL(__hip_atomic_load(tmo, RLX_AGENT)) == call || __builtin_amdgcn_s_memrealtime() - t0 > DEADLINE_TICKS) {
        __hip_atomic_store(tmo, call, RLX_AGENT);
        return true;
    }
    return false;
}
__device__ __forceinline__ unsigned umax(unsigned a, unsigned b) { return a > b ? a : b; }
__device__ __forceinline__ unsigned umin(unsigned a, unsigned b) { return a < b ? a : b; }
__device__ __forceinline__ unsigned wave_umax(unsigned v) {
    v = umax(v, (unsigned)__builtin_amdgcn_update_dpp(0, (int)v, 0xB1, 0xF, 0xF, true));
    v = umax(v, (unsigned)__builtin_amdgcn_update_dpp(0, (int)v, 0x4E, 0xF, 0xF, true));
    v = umax(v, (unsigned)__builtin_amdgcn_update_dpp(0, (int)v, 0x141, 0xF, 0xF, true));
    v = umax(v, (unsigned)__builtin_amdgcn_update_dpp(0, (int)v, 0x140, 0xF, 0xF, true));
    return umax(umax(__builtin_amdgcn_readlane(v, 0), __builtin_amdgcn_readlane(v, 16)),
                umax(__builtin_amdgcn_readlane(v, 32), __builtin_amdgcn_readlane(v, 48)));
}
#endif

__global__ __launch_bounds__(THREADS, 2) void gemm_a2a_kernel(Params p) {
#if defined(__HIP_DEVICE_COMPILE__)
    extern __shared__ __attribute__((aligned(16))) char shm_raw[];
    LAS char* shm = (LAS char*)shm_raw;
    const int b = blockIdx.x;
    const int grp = b & 7, idx = b >> 3;
    const int tn = idx & 3, tm = (idx >> 2) & 3, hb = idx >> 4;
    const int wid = RFL(threadIdx.x >> 6);
    const int lane = threadIdx.x & 63;
    GAS unsigned char* myws = (GAS unsigned char*)p.ws[p.rank & 7];
    GAS unsigned* tmo = (GAS unsigned*)(myws + WS_TMO);
    GAS unsigned* cnt = (GAS unsigned*)(myws + WS_CNT) + b;
    STAMP(0);

    volatile LAS unsigned* fl = (volatile LAS unsigned*)(shm + FL_OFF);
    auto r2_off = [&](int c, int k) { return (c * 2 + (k >> 3)) * STAGE_B + 8192 + (k & 7) * 1024; };
    auto r3_off = [&](int c, int k) { return c * STAGE_B + W_OFF + k * 1024; };

    f32x4 fin[8];
    const int wr = (wid >> 1) & 1, wc = wid & 1;
    unsigned char* pw = nullptr;

    if (wid >= 4) {
        if (LOADER_PRIO) __builtin_amdgcn_s_setprio(LOADER_PRIO);
        const int lw = wid - 4;
        const bool isw = lw < 2;
        auto rs = __builtin_amdgcn_make_buffer_rsrc((void*)((const char*)(isw ? p.w : p.x) - 4096), 0,
                                                    (isw ? KK * NN * 2 : ML * KK * 2) + 4096, 0x00020000);
        int vo[8];
#pragma unroll
        for (int i = 0; i < 8; ++i) {
            const int j = (lw & 1) * 8 + i;
            const int ow = (j * 4 + (lane >> 4)) * (NN * 2) + (grp * NL + tn * 128) * 2 + (((lane & 15) ^ (((lane >> 4) << 2) | (i & 3))) << 4);
            const int ox = (tm * 128 + j * 8 + (lane >> 3)) * (KK * 2) + (((lane & 7) ^ (((i & 1) << 2) | (lane >> 4))) << 4);
            vo[i] = (isw ? ow : ox) - (i & 3) * 1024 + 4096;
        }
        const int lds0 = (isw ? W_OFF : 0) + (lw & 1) * 8192, shl = isw ? 12 : 0;
        auto issue4 = [&](int s, int so, int g) {
            LAS char* dst = shm + (s & (ST - 1)) * STAGE_B + lds0 + g * 4096;
            __builtin_amdgcn_raw_ptr_buffer_load_lds(rs, dst, 16, vo[g * 4 + 0], so, 0, LOAD_AUX);
            __builtin_amdgcn_raw_ptr_buffer_load_lds(rs, dst, 16, vo[g * 4 + 1], so, 1024, LOAD_AUX);
            __builtin_amdgcn_raw_ptr_buffer_load_lds(rs, dst, 16, vo[g * 4 + 2], so, 2048, LOAD_AUX);
            __builtin_amdgcn_raw_ptr_buffer_load_lds(rs, dst, 16, vo[g * 4 + 3], so, 3072, LOAD_AUX);
        };
        issue4(0, (hb * NP1 * (BK * 2)) << shl, 0);
        issue4(0, (hb * NP1 * (BK * 2)) << shl, 1);
        unsigned c0 = 0;
        if (lw == 3) {
            c0 = __hip_atomic_load(cnt, RLX_AGENT);
            pw = p.ws[lane >> 3];
        }
        int ord;
        {
            const bool p2 = lane >= NP1;
            const int t = p2 ? lane - NP1 : lane, nt = p2 ? NP2 : NP1, lead = p2 ? LEAD2 : LEAD1, sh = p2 ? 3 : 2;
            const int q = p2 ? ((tm * 2 + hb + 2 * tn) & 7) : ((tm + tn) & 3);
            auto led_upto = [&](int m) { return m >= q ? ((m - q) >> sh) + 1 : 0; };
            const bool led = (t & ((1 << sh) - 1)) == q;
            int m = t - lead - 1;
            if (!p2 && m < 0) m = 0;
            const int hi = t + lead < nt - 1 ? t + lead : nt - 1;
            int pos = led ? ((t - q) >> sh) + (m + 1 - led_upto(m)) * (m >= 0) : t - led_upto(t) + led_upto(hi);
            if (!p2 && t == 0) pos = 0;
            pos = lane < NS ? pos + (p2 ? NP1 : 0) : lane;
            const int so = (((t + (p2 ? 2 * NP1 : hb * NP1)) * (BK * 2)) << shl) + ((p2 && !isw) ? hb * 64 * (KK * 2) : 0);
            ord = __builtin_amdgcn_ds_permute(pos << 2, so);
        }
#define VMW(n) asm volatile("s_waitcnt vmcnt(%0)" :: "i"(n) : "memory")
        auto issue8 = [&](int s) {
            const int so = __builtin_amdgcn_readlane(ord, s);
            issue4(s, so, 0);
            issue4(s, so, 1);
        };
        issue8(1);
        VMW(8);
        BARRIER();
        if (DIST == 3) issue8(2);
        const int nst = (lw == 3) ? NP1 : NS;
        issue8(DIST);
        VMW(8 * (DIST - 1));
        for (int s = 1; s < nst - DIST; ++s) {
            BARRIER();
            issue8(s + DIST);
            VMW(8 * (DIST - 1));
        }
        if (DIST == 3) { BARRIER(); VMW(8); }
        BARRIER();
        VMW(0);
        if (lw != 3) {
            BARRIER();
        } else {
            __builtin_amdgcn_s_setprio(0);
            BARRIER();
            const unsigned call = RFL(c0) + 1;
            fl[F_CALL] = call;
            fl[F_BAD] = 0;
            asm volatile("" : "+v"(pw));
            WAIT_LGKM0();
            GAS u64* xf = (GAS u64*)(myws + WS_XFLAG) + (b ^ 128);
            auto rp = __builtin_amdgcn_make_buffer_rsrc((void*)(p.ws[p.rank & 7] + WS_XCH + (size_t)(b ^ 128) * 32768), 0, 32768, 0x00020000);
            auto fetch = [&]() {
#pragma unroll
                for (int i = 0; i < 32; ++i)
                    __builtin_amdgcn_raw_ptr_buffer_load_lds(rp, shm + r2_off(i >> 4, i & 15), 16, lane * 16, i * 1024, 0, AUX_SYS);
            };
            auto both = [&](u64 v) { return RFL((int)((unsigned)v >= call && (unsigned)(v >> 32) >= call)) != 0; };
            bool got = false;
            for (int u = 0; u <= POLL_SLOT; ++u) BARRIER();
            constexpr int NPOLL = (NP2 - 2 - POLL_SLOT) / 2;
#pragma unroll
            for (int k = 0; k < NPOLL; ++k) {
                if (!got) {
                    u64 f;
                    asm volatile("global_load_dwordx2 %0, %1, off sc0 sc1\n\ts_barrier\n\ts_barrier\n\ts_waitcnt vmcnt(0)"
                                 : "=&v"(f) : "v"(xf) : "memory");
                    if (both(f)) { STAMP(8); fetch(); got = true; }
                } else {
                    BARRIER(); BARRIER();
                }
            }
            for (int u = POLL_SLOT + 1 + 2 * NPOLL; u < NP2 - 1; ++u) BARRIER();
            bool ok = true;
            if (!got) {
                const u64 t0 = __builtin_amdgcn_s_memrealtime();
                unsigned polls = 0;
                while (!both(__hip_atomic_load(xf, RLX_SYS)))
                    if (give_up(tmo, call, t0, polls)) { ok = false; break; }
                STAMP(8);
                if (ok) fetch();
            }
            WAIT_VM(0);
            fl[F_R2] = ok ? 1 : 2;
            WAIT_LGKM0();
            STAMP(9);
            BARRIER();
        }
    } else {
        const int qq = (lane & 15) >> 2, pp = lane & 3;
        const int gq = lane >> 4;
        int xo[2], wo[4][2];
#pragma unroll
        for (int ks = 0; ks < 2; ++ks) xo[ks] = (lane & 15) * 128 + (((4 * ks + gq) ^ ((lane >> 1) & 7)) << 4);
#pragma unroll
        for (int j = 0; j < 4; ++j)
#pragma unroll
            for (int t = 0; t < 2; ++t)
                wo[j][t] = W_OFF + 256 * (8 * gq + 4 * t + qq) +
                           (((2 * (wc * 4 + j) + (pp >> 1)) ^ ((qq << 2) | ((2 * gq + t) & 3))) << 4) + 8 * (pp & 1);
        f32x4 acc[4][4];
#define ACC_CHUNK(k) acc[(k) >> 2][(k) & 3]
#pragma unroll
        for (int k = 0; k < 16; ++k) ACC_CHUNK(k) = (f32x4){0.f, 0.f, 0.f, 0.f};
#define PIN_ADDR() asm volatile("" : "+v"(xo[0]), "+v"(xo[1]), "+v"(wo[0][0]), "+v"(wo[0][1]), "+v"(wo[1][0]), "+v"(wo[1][1]), \
                                     "+v"(wo[2][0]), "+v"(wo[2][1]), "+v"(wo[3][0]), "+v"(wo[3][1]))
        struct Frag { bf16x8 x[4], w[4]; };
        auto rd = [&](Frag& f, LAS char* sb, int xrow, int ks, bool full) {
#pragma unroll
            for (int j = 0; j < 4; ++j) {
                if (full || j < 2) f.x[j] = *(LAS bf16x8*)(sb + xrow + xo[ks] + j * 2048);
                LAS char* wb = sb + ks * 8192;
                const int jj = j;
                const bf16x4 lo = __builtin_amdgcn_ds_read_tr16_b64_v4bf16((LAS bf16x4*)(wb + wo[jj][0]));
                const bf16x4 hi = __builtin_amdgcn_ds_read_tr16_b64_v4bf16((LAS bf16x4*)(wb + wo[jj][1]));
                f.w[j] = __builtin_shufflevector(lo, hi, 0, 1, 2, 3, 4, 5, 6, 7);
            }
        };
        auto mm1 = [&](const Frag& f) {
#pragma unroll
            for (int i = 0; i < 4; ++i)
#pragma unroll
                for (int j = 0; j < 4; ++j)
                    acc[i][j] = __builtin_amdgcn_mfma_f32_16x16x32_bf16(f.w[j], f.x[i], acc[i][j], 0, 0, 0);
        };
        auto mm2 = [&](const Frag& f) {
#pragma unroll
            for (int i = 0; i < 2; ++i)
#pragma unroll
                for (int j = 0; j < 4; ++j)
                    fin[i * 4 + j] = __builtin_amdgcn_mfma_f32_16x16x32_bf16(f.w[j], f.x[i], fin[i * 4 + j], 0, 0, 0);
        };
#define SGB(m, n) __builtin_amdgcn_sched_group_barrier(m, n, 0)
#define SCHED_FENCE() __builtin_amdgcn_sched_barrier(0)
#define GROUP1_END() do { _Pragma("unroll") for (int _i = 0; _i < 12; ++_i) { SGB(0x8, 1); SGB(0x100, 1); } SGB(0x8, 4); SCHED_FENCE(); } while (0)
#define GROUP2_END() do { _Pragma("unroll") for (int _i = 0; _i < 5; ++_i) { SGB(0x8, 1); SGB(0x100, 2); } SGB(0x8, 3); SCHED_FENCE(); } while (0)
        auto stage_barrier = [&](int s) {
            WAIT_LGKM0();
            BARRIER();
            SCHED_FENCE();
        };
        Frag fa, fb;
        PIN_ADDR();
        stage_barrier(0);
        {
            STAMP(11);
            rd(fa, shm, wr * 8192, 0, true);
            SCHED_FENCE();
            rd(fb, shm, wr * 8192, 1, true); mm1(fa); GROUP1_END();
        }
        for (int s = 1; s < NP1; ++s) {
            stage_barrier(s);
            LAS char* sb = shm + (s & (ST - 1)) * STAGE_B;
            rd(fa, sb, wr * 8192, 0, true); mm1(fb); GROUP1_END();
            rd(fb, sb, wr * 8192, 1, true); mm1(fa); GROUP1_END();
        }
        stage_barrier(NP1);
        rd(fa, shm + (NP1 & (ST - 1)) * STAGE_B, wr * 4096, 0, false);
        mm1(fb);
        SCHED_FENCE();
        STAMP(1);
        const bool giver = (wr != hb);
        if (giver) {
#pragma unroll
            for (int k = 0; k < 8; ++k) fin[k] = (f32x4){0.f, 0.f, 0.f, 0.f};
        } else {
#pragma unroll
            for (int k = 0; k < 8; ++k)
                *(LAS f32x4*)(shm + KB_OFF + wc * 8192 + k * 1024 + lane * 16) = wr ? ACC_CHUNK(k) : ACC_CHUNK(8 + k);
#pragma unroll
            for (int k = 0; k < 8; ++k) fin[k] = wr ? ACC_CHUNK(8 + k) : ACC_CHUNK(k);
        }
        auto rdump = __builtin_amdgcn_make_buffer_rsrc((void*)(p.ws[p.rank & 7] + WS_XCH + (size_t)b * 32768), 0, 32768, 0x00020000);
        SCHED_FENCE();
#pragma unroll
        for (int u = 0; u < NP2; ++u) {
            LAS char* sb = shm + ((NP1 + u) & (ST - 1)) * STAGE_B;
            if (u > 0) { stage_barrier(NP1 + u); rd(fa, sb, wr * 4096, 0, false); mm2(fb); GROUP2_END(); }
            rd(fb, sb, wr * 4096, 1, false); mm2(fa); GROUP2_END();
            if (giver && u * DUMP_PER_STEP < 16) {
#pragma unroll
                for (int k = u * DUMP_PER_STEP; k < (u + 1) * DUMP_PER_STEP && k < 16; ++k)
                    __builtin_amdgcn_raw_buffer_store_b128(__builtin_bit_cast(u32x4, ACC_CHUNK(k)), rdump, lane * 16, wc * 16384 + k * 1024, AUX_SYS);
                asm volatile("s_nop 1" ::: "memory");
                SCHED_FENCE();
            }
            if (giver && u == FLAG_SLOT) {
                const unsigned call = RFL(fl[F_CALL]);
                WAIT_VM(0);
                if (lane == 0) __hip_atomic_store((GAS unsigned*)(myws + WS_XFLAG) + b * 2 + wc, call, RLX_SYS);
                STAMP(10);
                SCHED_FENCE();
            }
        }
        f32x4 c[8];
#pragma unroll
        for (int k = 0; k < 8; ++k) c[k] = *(LAS f32x4*)(shm + r2_off(wc, wr * 8 + k) + lane * 16);
        if (giver) {
#pragma unroll
            for (int k = 0; k < 8; ++k) c[k] += *(LAS f32x4*)(shm + KB_OFF + wc * 8192 + k * 1024 + lane * 16);
        }
        mm2(fb);
        STAMP(2);
        unsigned mx = 0;
#pragma unroll
        for (int k = 0; k < 8; ++k)
#pragma unroll
            for (int e = 0; e < 4; ++e) {
                fin[k][e] = fmaxf(fin[k][e] + c[k][e], 0.f);
                mx = umax(mx, __float_as_uint(fin[k][e]) & 0x7fffffffu);
            }
        fl[F_WMAX + wid] = wave_umax(mx);
        WAIT_LGKM0();
    }
    BARRIER();
    STAMP(3);
    const unsigned call = RFL(fl[F_CALL]);
    if (wid < 4) {
#pragma unroll
        for (int k = 0; k < 8; ++k)
            *(LAS f32x4*)(shm + r3_off(wc, wr * 8 + k) + ((lane ^ (((MF16 ? k : (k >> 1)) & 3) << 2)) << 4)) = fin[k];
        WAIT_LGKM0();
    } else if (wid == 7) {
        const u32x4 m4 = *(volatile LAS u32x4*)(shm + FL_OFF + F_WMAX * 4);
        const u64 gran = ((u64)call << 32) | umax(umax(m4[0], m4[1]), umax(m4[2], m4[3]));
        if (lane == 0) __hip_atomic_store((GAS u64*)(myws + WS_LMAX) + b, gran, RLX_AGENT);
    }
    BARRIER();
    f32x4 y[4];
    {
        const int m = wid * 8 + (lane >> 3), cg = lane & 7;
#pragma unroll
        for (int i = 0; i < 4; ++i)
            y[i] = *(LAS f32x4*)(shm + r3_off(cg >> 2, (m >> 4) * 4 + (cg & 3)) + ((((m & 15) + 16 * i) ^ ((cg & 3) << 2)) << 4));
    }
    if (wid == 7) {
        const u64 t0 = __builtin_amdgcn_s_memrealtime();
        unsigned polls = 0;
        const size_t goff = WS_GMAX + (size_t)(call & 1) * (NREP * 128);
        if (b == 0) {
            GAS u64* lmax = (GAS u64*)(myws + WS_LMAX);
            unsigned gm;
            bool bad = false;
            for (;;) {
                bool ok = true;
                gm = 0;
#pragma unroll
                for (int k = 0; k < 4; ++k) {
                    const u64 v = __hip_atomic_load(lmax + k * 64 + lane, RLX_AGENT);
                    ok &= (unsigned)(v >> 32) >= call;
                    gm = umax(gm, (unsigned)v);
                }
                if (__all(ok)) break;
                if (give_up(tmo, call, t0, polls)) { bad = true; break; }
            }
            gm = wave_umax(gm);
            if (!bad)
                __hip_atomic_store((GAS u64*)(pw + goff + (lane & 7) * 128) + SRC_SLOT(lane >> 3), ((u64)call << 32) | gm, RLX_SYS);
            STAMP(4);
        }
        GAS u64* gp = lane < NR ? (GAS u64*)(myws + goff + grp * 128) + lane : (GAS u64*)tmo;
        unsigned am;
        bool bad = fl[F_R2] != 1u;
        for (;;) {
            const u64 v = __hip_atomic_load(gp, RLX_SYS);
            am = lane < NR ? (unsigned)v : 0u;
            if (__any(lane >= NR && (unsigned)v == call)) { bad = true; break; }
            if (__all(lane >= NR || (unsigned)(v >> 32) >= call)) break;
            if (give_up(tmo, call, t0, polls)) { bad = true; break; }
        }
        fl[F_AMAX] = wave_umax(am);
        fl[F_BAD] = bad;
        WAIT_LGKM0();
    }
    BARRIER();
    STAMP(5);
    {
        const float amax = __uint_as_float(fl[F_AMAX]);
        const unsigned bad = fl[F_BAD];
        const float sc = amax / 448.0f;
        const float inv = sc > 0.f ? 1.0f / sc : 0.f;
        u32x4 o;
#pragma unroll
        for (int i = 0; i < 4; ++i) {
            int w = __builtin_amdgcn_cvt_pk_fp8_f32(y[i][0] * inv, y[i][1] * inv, 0, false);
            w = __builtin_amdgcn_cvt_pk_fp8_f32(y[i][2] * inv, y[i][3] * inv, w, true);
            o[i] = bad ? 0x7f7f7f7fu : (unsigned)w;
        }
        const int row = (p.rank & 7) * ML + tm * 128 + hb * 64 + wid * 8 + (lane >> 3);
        auto ro = __builtin_amdgcn_make_buffer_rsrc((void*)p.out[grp], 0, NR * ML * NL, 0x00020000);
        __builtin_amdgcn_raw_buffer_store_b128(o, ro, row * NL + tn * 128 + (lane & 7) * 16, 0, AUX_SYS);
    }
    WAIT_VM(0);
    BARRIER();
    STAMP(6);
    if (wid == 7) {
        if (lane == 0) {
            __hip_atomic_store((GAS unsigned*)((GAS unsigned char*)p.ws[grp] + WS_DONE) + SRC_SLOT(grp) * 32 + idx, call, RLX_SYS);
            __hip_atomic_store(cnt, call, RLX_AGENT);
        }
        if (b == 0) {
            GAS u64* dn = (GAS u64*)(myws + WS_DONE);
            const u64 t0 = __builtin_amdgcn_s_memrealtime();
            unsigned polls = 0;
            for (;;) {
                const u64 v0 = __hip_atomic_load(dn + lane, RLX_SYS), v1 = __hip_atomic_load(dn + 64 + lane, RLX_SYS);
                const bool ok = (unsigned)v0 >= call && (unsigned)(v0 >> 32) >= call && (unsigned)v1 >= call && (unsigned)(v1 >> 32) >= call;
                if (__all(ok)) break;
                if (give_up(tmo, call, t0, polls)) {
                    if (lane == 0)
                        __hip_atomic_store((GAS unsigned*)((GAS unsigned char*)p.out[p.rank & 7] + (size_t)(p.rank & 7) * ML * NL), 0x7f7f7f7fu, RLX_SYS);
                    break;
                }
            }
            STAMP(7);
        }
    }
#endif
}

}

extern "C" void kernel_launch(int rank, int nranks, ncclComm_t comm, hipStream_t stream,
                              void* const* d_in, const int* in_sizes, int n_in,
                              void* d_out, int out_size, const int* in_dtypes, int out_dtype,
                              void* const* d_out_all, void* const* d_ws_all, size_t ws_bytes) {
    (void)comm; (void)in_dtypes; (void)out_dtype; (void)d_out;
    if (nranks != ga::NR || (unsigned)rank >= (unsigned)ga::NR || n_in < 2 || in_sizes[0] != ga::ML * ga::KK ||
        in_sizes[1] != ga::KK * ga::NN || out_size != ga::NR * ga::ML * ga::NL || ws_bytes < ga::WS_END) {
        fprintf(stderr, "kernel_launch: unexpected problem shape\n");
        return;
    }
    HIP_CHECK(hipFuncSetAttribute((const void*)ga::gemm_a2a_kernel, hipFuncAttributeMaxDynamicSharedMemorySize, ga::LDS_BYTES));
    ga::Params p;
    p.x = d_in[0];
    p.w = d_in[1];
    for (int r = 0; r < ga::NR; ++r) {
        p.out[r] = (unsigned char*)d_out_all[r];
        p.ws[r] = (unsigned char*)d_ws_all[r];
    }
    p.rank = rank;
    p.pad = 0;
    hipLaunchKernelGGL(ga::gemm_a2a_kernel, dim3(ga::GRID), dim3(ga::THREADS), ga::LDS_BYTES, stream, p);
    HIP_CHECK(hipGetLastError());
}
```

```cpp
#include <hip/hip_runtime.h>
#include <rccl/rccl.h>
#include <cstdio>

#define HIP_CHECK(expr) do { hipError_t _e = (expr); if (_e != hipSuccess) { \
    fprintf(stderr, "HIP error %s at %s:%d\n", hipGetErrorString(_e), __FILE__, __LINE__); \
    return; } } while (0)

#define NP2_STEPS 12
#define FLAG_SLOT_ 4
#define POLL_SLOT_ 7
#define XBF16 1
#define XPS_ 8
#define LOADER_PRIO 0
#define LEAD1 3
#define LEAD2 4
#define MM_ORDER 0
#define OUT_AUX 17
#define WMAP 0
#define KROT1 0
#define KROT2 0
#define START_MODE 0
#define NCOLL 1
#define RAMP_ 4
#define LOAD_AUX 0

namespace ga {

#define LAS __attribute__((address_space(3)))
#define GAS __attribute__((address_space(1)))
typedef __bf16 bf16x4 __attribute__((ext_vector_type(4)));
typedef __bf16 bf16x8 __attribute__((ext_vector_type(8)));
typedef float f32x4 __attribute__((ext_vector_type(4)));
typedef unsigned u32x4 __attribute__((ext_vector_type(4)));
typedef unsigned long long u64;

constexpr int NR = 8;
constexpr int ML = 512, NN = 4096, KK = 4096, NL = 512;
constexpr int BK = 64, NKS = KK / BK;
constexpr int NP2 = NP2_STEPS;
constexpr int NP1 = (NKS - NP2) / 2;
constexpr int NS = NP1 + NP2;
static_assert(2 * NP1 + NP2 == NKS && NP2 >= 6, "K split");
constexpr int ST = 4, STAGE_B = 32768, W_OFF = 16384;
constexpr int KB_OFF = ST * STAGE_B;
constexpr int FL_OFF = KB_OFF + 16384;
constexpr int LDS_BYTES = FL_OFF + 256;
constexpr int F_WMAX = 8, F_AMAX = 16, F_CALL = 18, F_R2 = 19, F_N = 24;
constexpr int GRID = 256, THREADS = 512, RAMP = RAMP_;
static_assert(RAMP + 4 <= NP1, "prefetch ramp");
constexpr int FLAG_SLOT = FLAG_SLOT_, POLL_SLOT = POLL_SLOT_;
constexpr int NXB = XBF16 ? 16 : 32, XPS = XPS_;
static_assert(NXB % XPS == 0 && FLAG_SLOT >= NXB / XPS && POLL_SLOT >= FLAG_SLOT && POLL_SLOT + 2 + NXB / XPS <= NP2, "hand-off schedule");

constexpr size_t WS_CNT = 0x0, WS_TMO = 0x1000, WS_LMAX = 0x2000, WS_GMAX = 0x4000,
                 WS_XFLAG = 0x8000, WS_STAMP = 0x10000, WS_XCH = 0x100000,
                 WS_MASK = WS_XCH + (size_t)GRID * 32768,
                 WS_PAY = WS_MASK + 2 * (size_t)GRID * 2048,
                 WS_END = WS_PAY + (size_t)GRID * 8192;
constexpr int NREP = 8;
constexpr u64 DEADLINE_TICKS = 50000000ull;

struct Params {
    const void* x;
    const void* w;
    unsigned char* out[NR];
    unsigned char* ws[NR];
    int rank;
    int pad;
};

#define RLX_AGENT __ATOMIC_RELAXED, __HIP_MEMORY_SCOPE_AGENT
#define RLX_SYS __ATOMIC_RELAXED, __HIP_MEMORY_SCOPE_SYSTEM
#define BARRIER() asm volatile("s_barrier" ::: "memory")
#define WAIT_VM(n) asm volatile("s_waitcnt vmcnt(" #n ")" ::: "memory")
#define WAIT_LGKM0() asm volatile("s_waitcnt lgkmcnt(0)" ::: "memory")
#define RFL(x) __builtin_amdgcn_readfirstlane(x)

#define SRC_SLOT(d) (p.rank & 7)
#define OUT_RANK (p.rank & 7)
#define OUT_ROW0 (grp * ML)
#define STAMP(i) do {} while (0)

#if defined(__HIP_DEVICE_COMPILE__)
constexpr int AUX_SYS = 17;
__device__ __forceinline__ bool give_up(GAS unsigned* tmo, unsigned call, u64 t0, unsigned& polls) {
    if ((++polls & 31u) != 0u) return false;
    if (RFL(__hip_atomic_load(tmo, RLX_AGENT)) == call || __builtin_amdgcn_s_memrealtime() - t0 > DEADLINE_TICKS) {
        __hip_atomic_store(tmo, call, RLX_AGENT);
        return true;
    }
    return false;
}
typedef float f32x2 __attribute__((ext_vector_type(2)));
typedef __bf16 bf16x2 __attribute__((ext_vector_type(2)));
__device__ __forceinline__ unsigned pack2(float a, float b) {
    return __builtin_bit_cast(unsigned, __builtin_convertvector((f32x2){a, b}, bf16x2));
}
__device__ __forceinline__ u32x4 pack_bf16(f32x4 a, f32x4 b) {
    return (u32x4){pack2(a[0], a[1]), pack2(a[2], a[3]), pack2(b[0], b[1]), pack2(b[2], b[3])};
}
__device__ __forceinline__ unsigned umax(unsigned a, unsigned b) { return a > b ? a : b; }
__device__ __forceinline__ unsigned umin(unsigned a, unsigned b) { return a < b ? a : b; }
__device__ __forceinline__ unsigned wave_scan(unsigned v) {
    v += (unsigned)__builtin_amdgcn_update_dpp(0, (int)v, 0x111, 0xF, 0xF, true);
    v += (unsigned)__builtin_amdgcn_update_dpp(0, (int)v, 0x112, 0xF, 0xF, true);
    v += (unsigned)__builtin_amdgcn_update_dpp(0, (int)v, 0x114, 0xF, 0xF, true);
    v += (unsigned)__builtin_amdgcn_update_dpp(0, (int)v, 0x118, 0xF, 0xF, true);
    v += (unsigned)__builtin_amdgcn_update_dpp(0, (int)v, 0x142, 0xA, 0xF, false);
    v += (unsigned)__builtin_amdgcn_update_dpp(0, (int)v, 0x143, 0xC, 0xF, false);
    return v;
}
__device__ __forceinline__ unsigned spread4(unsigned nib) { return (nib * 0x00204081u) & 0x01010101u; }
__device__ __forceinline__ unsigned wave_umax(unsigned v) {
    v = umax(v, (unsigned)__builtin_amdgcn_update_dpp(0, (int)v, 0xB1, 0xF, 0xF, true));
    v = umax(v, (unsigned)__builtin_amdgcn_update_dpp(0, (int)v, 0x4E, 0xF, 0xF, true));
    v = umax(v, (unsigned)__builtin_amdgcn_update_dpp(0, (int)v, 0x141, 0xF, 0xF, true));
    v = umax(v, (unsigned)__builtin_amdgcn_update_dpp(0, (int)v, 0x140, 0xF, 0xF, true));
    return umax(umax(__builtin_amdgcn_readlane(v, 0), __builtin_amdgcn_readlane(v, 16)),
                umax(__builtin_amdgcn_readlane(v, 32), __builtin_amdgcn_readlane(v, 48)));
}
#endif

__global__ __launch_bounds__(THREADS, 2) void gemm_a2a_kernel(Params p) {
#if defined(__HIP_DEVICE_COMPILE__)
    extern __shared__ __attribute__((aligned(16))) char shm_raw[];
    LAS char* shm = (LAS char*)shm_raw;
    const int b = blockIdx.x;
    const int grp = b & 7, idx = b >> 3;
    const int tn = idx & 3, tm = (idx >> 2) & 3, hb = idx >> 4;
    const int wid = RFL(threadIdx.x >> 6);
    const int lane = threadIdx.x & 63;
    GAS unsigned char* myws = (GAS unsigned char*)p.ws[p.rank & 7];
    GAS unsigned* tmo = (GAS unsigned*)(myws + WS_TMO);
    GAS unsigned* cnt = (GAS unsigned*)(myws + WS_CNT) + b;
    STAMP(0);

    volatile LAS unsigned* fl = (volatile LAS unsigned*)(shm + FL_OFF);
    auto x2_off = [&](int i) { return (i >> 3) * STAGE_B + 8192 + (i & 7) * 1024; };
    auto r3_off = [&](int c, int k) { return c * STAGE_B + W_OFF + k * 1024; };

    f32x4 fin[8];
    const int wr = (wid >> 1) & 1, wc = wid & 1;
    unsigned char* pw = nullptr;

    if (wid >= 4) {
        if (LOADER_PRIO) __builtin_amdgcn_s_setprio(LOADER_PRIO);
        const int lw = wid - 4;
        const bool isw = lw < 2;
        auto rs = __builtin_amdgcn_make_buffer_rsrc((void*)((const char*)(isw ? p.w : p.x) - 4096), 0,
                                                    (isw ? KK * NN * 2 : ML * KK * 2) + 4096, 0x00020000);
        int vo[8];
#pragma unroll
        for (int i = 0; i < 8; ++i) {
            const int j = (lw & 1) * 8 + i;
            const int ow = (j * 4 + (lane >> 4)) * (NN * 2) + (grp * NL + tn * 128) * 2 + (((lane & 15) ^ (((lane >> 4) << 2) | (i & 3))) << 4);
            const int ox = (tm * 128 + j * 8 + (lane >> 3)) * (KK * 2) + (((lane & 7) ^ (((i & 1) << 2) | (lane >> 4))) << 4);
            vo[i] = (isw ? ow : ox) - (i & 3) * 1024 + 4096;
        }
        const int lds0 = (isw ? W_OFF : 0) + (lw & 1) * 8192, shl = isw ? 12 : 0;
        auto issue4 = [&](int s, int so, int g) {
            LAS char* dst = shm + (s & (ST - 1)) * STAGE_B + lds0 + g * 4096;
            __builtin_amdgcn_raw_ptr_buffer_load_lds(rs, dst, 16, vo[g * 4 + 0], so, 0, LOAD_AUX);
            __builtin_amdgcn_raw_ptr_buffer_load_lds(rs, dst, 16, vo[g * 4 + 1], so, 1024, LOAD_AUX);
            __builtin_amdgcn_raw_ptr_buffer_load_lds(rs, dst, 16, vo[g * 4 + 2], so, 2048, LOAD_AUX);
            __builtin_amdgcn_raw_ptr_buffer_load_lds(rs, dst, 16, vo[g * 4 + 3], so, 3072, LOAD_AUX);
        };
        const int rot1 = (grp * KROT1) % NP1, t0s = START_MODE == 1 ? ((tm + tn) & 3) : 0;
        issue4(0, ((hb * NP1 + (rot1 + t0s) % NP1) * (BK * 2)) << shl, 0);
        issue4(0, ((hb * NP1 + (rot1 + t0s) % NP1) * (BK * 2)) << shl, 1);
        unsigned c0 = 0;
        if (lw == 3) {
            c0 = __hip_atomic_load(cnt, RLX_AGENT);
            pw = p.ws[lane >> 3];
        }
        int ord;
        {
            const bool p2 = lane >= NP1;
            const int t = p2 ? lane - NP1 : lane, nt = p2 ? NP2 : NP1, lead = p2 ? LEAD2 : LEAD1, sh = p2 ? 3 : 2;
            const int q = p2 ? ((tm * 2 + hb + 2 * tn) & 7) : ((tm + tn) & 3);
            auto cls_upto = [&](int m) { return m >= q ? ((m - q) >> sh) + 1 : 0; };
            auto led_upto = [&](int m) { return m > lead ? cls_upto(m) - cls_upto(lead) : 0; };
            const bool led = (t & ((1 << sh) - 1)) == q && t > lead;
            const int m = t - lead - 1, hi = t + lead < nt - 1 ? t + lead : nt - 1;
            int pos = led ? led_upto(t) - 1 + m + 1 - led_upto(m) : t - led_upto(t) + led_upto(hi);
            pos = lane < NS ? pos + (p2 ? NP1 : 0) : lane;
            int tr = t + (p2 ? (grp * KROT2) % NP2 : rot1);
            if (tr >= nt) tr -= nt;
            const int so = (((tr + (p2 ? 2 * NP1 : hb * NP1)) * (BK * 2)) << shl) + ((p2 && !isw) ? hb * 64 * (KK * 2) : 0);
            ord = __builtin_amdgcn_ds_permute(pos << 2, so);
        }
#define VMW(n) asm volatile("s_waitcnt vmcnt(%0)" :: "i"(n) : "memory")
        auto issue8 = [&](int s) {
            const int so = __builtin_amdgcn_readlane(ord, s);
            issue4(s, so, 0);
            issue4(s, so, 1);
        };
        issue8(1);
        VMW(8);
        for (int s = 0; s < RAMP; ++s) {
            BARRIER();
            issue8(s + 2);
            VMW(8);
        }
        const int nst = (lw == 3) ? NP1 : NS;
        BARRIER();
        issue8(RAMP + 2);
        issue8(RAMP + 3);
        VMW(16);
        for (int s = RAMP + 1; s < nst - 3; ++s) {
            BARRIER();
            issue8(s + 3);
            VMW(16);
        }
        BARRIER();
        VMW(8);
        BARRIER();
        VMW(0);
        if (lw != 3) {
            BARRIER();
        } else {
            __builtin_amdgcn_s_setprio(0);
            const unsigned call = RFL(c0) + 1;
            fl[F_CALL] = call;
            fl[F_AMAX + 1] = 0;
            fl[F_R2] = 0;
            asm volatile("" : "+v"(pw));
            GAS unsigned* xf = (GAS unsigned*)(myws + WS_XFLAG);
            auto rdump = __builtin_amdgcn_make_buffer_rsrc((void*)(p.ws[p.rank & 7] + WS_XCH + (size_t)b * 32768), 0, 32768, 0x00020000);
            auto rp = __builtin_amdgcn_make_buffer_rsrc((void*)(p.ws[p.rank & 7] + WS_XCH + (size_t)(b ^ 128) * 32768), 0, 32768, 0x00020000);
            auto fetch = [&](int i0, int i1) {
#pragma unroll
                for (int i = i0; i < i1; ++i)
                    __builtin_amdgcn_raw_ptr_buffer_load_lds(rp, shm + x2_off(i), 16, lane * 16, i * 1024, 0, AUX_SYS);
            };
            BARRIER();
            BARRIER();
            constexpr int NXS = NXB / XPS;
#pragma unroll
            for (int k = 0; k < NXS; ++k) {
                BARRIER();
                u32x4 v[XPS];
#pragma unroll
                for (int i = 0; i < XPS; ++i) v[i] = *(LAS u32x4*)(shm + x2_off(k * XPS + i) + lane * 16);
#pragma unroll
                for (int i = 0; i < XPS; ++i)
                    __builtin_amdgcn_raw_buffer_store_b128(v[i], rdump, lane * 16, (k * XPS + i) * 1024, AUX_SYS);
                asm volatile("s_nop 1" ::: "memory");
            }
            for (int u = NXS + 1; u <= FLAG_SLOT; ++u) BARRIER();
            WAIT_VM(0);
            if (lane == 0) __hip_atomic_store(xf + b, call, RLX_SYS);
            STAMP(10);
            for (int u = FLAG_SLOT + 1; u <= POLL_SLOT; ++u) BARRIER();
            unsigned f;
            asm volatile("global_load_dword %0, %1, off sc0 sc1\n\ts_barrier\n\ts_barrier\n\ts_waitcnt vmcnt(0)"
                         : "=&v"(f) : "v"(xf + (b ^ 128)) : "memory");
            const bool got = RFL(f) >= call;
            if (got) STAMP(8);
#pragma unroll
            for (int k = 0; k < NXS; ++k) {
                if (k > 0) BARRIER();
                if (got) fetch(k * XPS, (k + 1) * XPS);
            }
            for (int u = POLL_SLOT + 2 + NXS; u < NP2; ++u) BARRIER();
            bool ok = true;
            if (!got) {
                const u64 t0 = __builtin_amdgcn_s_memrealtime();
                unsigned polls = 0;
                while (RFL(__hip_atomic_load(xf + (b ^ 128), RLX_SYS)) < call)
                    if (give_up(tmo, call, t0, polls)) { ok = false; break; }
                STAMP(8);
                if (ok) fetch(0, NXB);
            }
            WAIT_VM(0);
            fl[F_R2] = ok ? 1 : 2;
            WAIT_LGKM0();
            STAMP(9);
        }
    } else {
        const int gq = lane >> 4, qq = (lane & 15) >> 2, pp = lane & 3;
        int xo[2], wo[4][2];
#pragma unroll
        for (int ks = 0; ks < 2; ++ks) xo[ks] = (lane & 15) * 128 + (((4 * ks + gq) ^ ((lane >> 1) & 7)) << 4);
#pragma unroll
        for (int j = 0; j < 4; ++j)
#pragma unroll
            for (int t = 0; t < 2; ++t)
                wo[j][t] = W_OFF + 256 * (8 * gq + 4 * t + qq) +
                           (((2 * (wc * 4 + j) + (pp >> 1)) ^ ((qq << 2) | ((2 * gq + t) & 3))) << 4) + 8 * (pp & 1);
        f32x4 acc[4][4];
#pragma unroll
        for (int k = 0; k < 16; ++k) acc[k >> 2][k & 3] = (f32x4){0.f, 0.f, 0.f, 0.f};
        struct Frag { bf16x8 x[4], w[4]; };
        auto rd = [&](Frag& f, LAS char* sb, int x0, int x1, int ks, bool full) {
#pragma unroll
            for (int j = 0; j < 4; ++j) {
                if (full || j < 2) f.x[j] = *(LAS bf16x8*)(sb + (j < 2 ? x0 : x1) + xo[ks] + (j & 1) * 2048);
                const bf16x4 lo = __builtin_amdgcn_ds_read_tr16_b64_v4bf16((LAS bf16x4*)(sb + ks * (WMAP ? 4096 : 8192) + wo[j][0]));
                const bf16x4 hi = __builtin_amdgcn_ds_read_tr16_b64_v4bf16((LAS bf16x4*)(sb + ks * (WMAP ? 4096 : 8192) + wo[j][1]));
                f.w[j] = __builtin_shufflevector(lo, hi, 0, 1, 2, 3, 4, 5, 6, 7);
            }
        };
        auto mm = [&](const Frag& f, int ni) {
#pragma unroll
            for (int i = 0; i < ni; ++i)
#pragma unroll
                for (int j = 0; j < 4; ++j)
                    acc[i][j] = __builtin_amdgcn_mfma_f32_16x16x32_bf16(f.w[j], f.x[i], acc[i][j], 0, 0, 0);
        };
#define SGB(m, n) __builtin_amdgcn_sched_group_barrier(m, n, 0)
#define SCHED_FENCE() __builtin_amdgcn_sched_barrier(0)
#define GROUP1_END() do { _Pragma("unroll") for (int _i = 0; _i < 12; ++_i) { SGB(0x8, 1); SGB(0x100, 1); } SGB(0x8, 4); SCHED_FENCE(); } while (0)
#define GROUP2_END() do { _Pragma("unroll") for (int _i = 0; _i < 5; ++_i) { SGB(0x8, 1); SGB(0x100, 2); } SGB(0x8, 3); SCHED_FENCE(); } while (0)
        auto stage_barrier = [&](int s) {
            WAIT_LGKM0();
            BARRIER();
            SCHED_FENCE();
        };
        Frag fa, fb;
        asm volatile("" : "+v"(xo[0]), "+v"(xo[1]), "+v"(wo[0][0]), "+v"(wo[0][1]), "+v"(wo[1][0]), "+v"(wo[1][1]),
                          "+v"(wo[2][0]), "+v"(wo[2][1]), "+v"(wo[3][0]), "+v"(wo[3][1]));
        stage_barrier(0);
        const int xa = wr * 12288, xb = 4096 + wr * 4096;
        {
            STAMP(11);
            rd(fa, shm, xa, xb, 0, true);
            SCHED_FENCE();
            rd(fb, shm, xa, xb, 1, true); mm(fa, 4); GROUP1_END();
        }
        for (int s = 1; s < NP1; ++s) {
            stage_barrier(s);
            LAS char* sb = shm + (s & (ST - 1)) * STAGE_B;
            rd(fa, sb, xa, xb, 0, true); mm(fb, 4); GROUP1_END();
            rd(fb, sb, xa, xb, 1, true); mm(fa, 4); GROUP1_END();
        }
        stage_barrier(NP1);
        rd(fa, shm + (NP1 & (ST - 1)) * STAGE_B, wr * 4096, 0, 0, false);
        mm(fb, 4);
        SCHED_FENCE();
        STAMP(1);
        const bool giver = (wr != hb);
        if (giver) {
#pragma unroll
            for (int k = 0; k < 8; ++k)
                *(LAS u32x4*)(shm + x2_off(wc * 8 + (((k >> 2) ^ wr) << 2)) + (k & 3) * 1024 + lane * 16) =
                    pack_bf16(acc[k >> 1][(k & 1) * 2], acc[k >> 1][(k & 1) * 2 + 1]);
#pragma unroll
            for (int k = 0; k < 8; ++k) acc[k >> 2][k & 3] = (f32x4){0.f, 0.f, 0.f, 0.f};
        } else {
#pragma unroll
            for (int k = 0; k < 8; ++k)
                *(LAS f32x4*)(shm + KB_OFF + wc * 8192 + k * 1024 + lane * 16) = acc[2 + (k >> 2)][k & 3];
        }
        SCHED_FENCE();
#pragma unroll
        for (int u = 0; u < NP2; ++u) {
            LAS char* sb = shm + ((NP1 + u) & (ST - 1)) * STAGE_B;
            if (u > 0) { stage_barrier(NP1 + u); rd(fa, sb, wr * 4096, 0, 0, false); mm(fb, 2); GROUP2_END(); }
            rd(fb, sb, wr * 4096, 0, 1, false); mm(fa, 2); GROUP2_END();
        }
        f32x4 c[8];
        if (giver) {
#pragma unroll
            for (int k = 0; k < 8; ++k) c[k] = *(LAS f32x4*)(shm + KB_OFF + wc * 8192 + k * 1024 + lane * 16);
        } else {
#pragma unroll
            for (int k = 0; k < 8; ++k) c[k] = (f32x4){0.f, 0.f, 0.f, 0.f};
        }
        mm(fb, 2);
        while (RFL(fl[F_R2]) == 0u) {}
        asm volatile("" ::: "memory");
#pragma unroll
        for (int k = 0; k < 4; ++k) {
            const u32x4 v = *(LAS u32x4*)(shm + x2_off(wc * 8 + wr * 4) + k * 1024 + lane * 16);
#pragma unroll
            for (int e = 0; e < 4; ++e) {
                c[2 * k + (e >> 1)][(e & 1) * 2] += __uint_as_float(v[e] << 16);
                c[2 * k + (e >> 1)][(e & 1) * 2 + 1] += __uint_as_float(v[e] & 0xffff0000u);
            }
        }
        STAMP(2);
        unsigned mx = 0;
#pragma unroll
        for (int k = 0; k < 8; ++k)
#pragma unroll
            for (int e = 0; e < 4; ++e) {
                fin[k][e] = fmaxf(acc[k >> 2][k & 3][e] + c[k][e], 0.f);
                mx = umax(mx, __float_as_uint(fin[k][e]) & 0x7fffffffu);
            }
        fl[F_WMAX + wid] = wave_umax(mx);
        WAIT_LGKM0();
    }
    BARRIER();
    STAMP(3);
    const unsigned call = RFL(fl[F_CALL]);
    if (wid < 4) {
#pragma unroll
        for (int k = 0; k < 8; ++k)
            *(LAS f32x4*)(shm + r3_off(wc, wr * 8 + k) + ((lane ^ ((k & 3) << 2)) << 4)) = fin[k];
        WAIT_LGKM0();
    } else if (wid == 7) {
        const u32x4 m4 = *(volatile LAS u32x4*)(shm + FL_OFF + F_WMAX * 4);
        const u64 gran = ((u64)call << 32) | umax(umax(m4[0], m4[1]), umax(m4[2], m4[3]));
        if (lane == 0) __hip_atomic_store((GAS u64*)(myws + WS_LMAX) + b, gran, RLX_AGENT);
    }
    BARRIER();
    f32x4 y[4];
    {
        const int m = wid * 8 + (lane >> 3), cg = lane & 7;
#pragma unroll
        for (int i = 0; i < 4; ++i)
            y[i] = *(LAS f32x4*)(shm + r3_off(cg >> 2, (m >> 4) * 4 + (cg & 3)) + ((((m & 15) + 16 * i) ^ ((cg & 3) << 2)) << 4));
    }
    const bool remote = grp != (p.rank & 7);
    const unsigned tagbits = spread4(call % 15u + 1u) << 7;
    auto cp = [&](int w) { return shm + 2 * STAGE_B + w * 8192; };
    auto ro = __builtin_amdgcn_make_buffer_rsrc((void*)p.out[OUT_RANK], 0, NR * ML * NL, 0x00020000);
    const int out0 = (OUT_ROW0 + tm * 128 + hb * 64 + (lane >> 3)) * NL + tn * 128 + (lane & 7) * 16;
    if (remote) {
        unsigned mask = 0;
#pragma unroll
        for (int e = 0; e < 16; ++e) mask |= (y[e >> 2][e & 3] > 0.f ? 1u : 0u) << e;
        const unsigned c = __builtin_popcount(mask), incl = wave_scan(c);
        unsigned pos = (incl - c) * 4;
#pragma unroll
        for (int e = 0; e < 16; ++e)
            if (y[e >> 2][e & 3] > 0.f) { *(LAS float*)(cp(wid) + pos) = y[e >> 2][e & 3]; pos += 4; }
        *(LAS unsigned*)(cp(wid) + 4096 + lane * 4) = mask | ((call % 65535u + 1u) << 16);
        fl[F_N + wid] = __builtin_amdgcn_readlane(incl, 63);
        WAIT_LGKM0();
        BARRIER();
    }
    if (wid == 7 || (NCOLL > 1 && wid >= 8 - NCOLL && b == (p.rank & 7))) {
        const u64 t0 = __builtin_amdgcn_s_memrealtime();
        unsigned polls = 0;
        const size_t goff = WS_GMAX + (size_t)(call & 1) * (NREP * 128);
        if (b == (p.rank & 7)) {
            if (NCOLL > 1) { pw = p.ws[lane >> 3]; asm volatile("" : "+v"(pw)); __builtin_amdgcn_s_sleep(0); for (int i = 0; i < (7 - wid); ++i) __builtin_amdgcn_s_sleep(16 / NCOLL); }
            GAS u64* lmax = (GAS u64*)(myws + WS_LMAX);
            unsigned gm;
            bool bad = false;
            for (;;) {
                bool ok = true;
                gm = 0;
#pragma unroll
                for (int k = 0; k < 4; ++k) {
                    const u64 v = __hip_atomic_load(lmax + k * 64 + lane, RLX_AGENT);
                    ok &= (unsigned)(v >> 32) >= call;
                    gm = umax(gm, (unsigned)v);
                }
                if (__all(ok)) break;
                if (give_up(tmo, call, t0, polls)) { bad = true; break; }
            }
            gm = wave_umax(gm);
            if (!bad)
                __hip_atomic_store((GAS u64*)(pw + goff + (lane & 7) * 128) + SRC_SLOT(lane >> 3), ((u64)call << 32) | gm, RLX_SYS);
            STAMP(4);
        }
        GAS u64* gp = lane < NR ? (GAS u64*)(myws + goff + grp * 128) + lane : (GAS u64*)tmo;
        unsigned am;
        bool bad = fl[F_R2] != 1u;
        for (;;) {
            const u64 v = __hip_atomic_load(gp, RLX_SYS);
            am = lane < NR ? (unsigned)v : 0u;
            if (__any(lane >= NR && (unsigned)v == call)) { bad = true; break; }
            if (__all(lane >= NR || (unsigned)(v >> 32) >= call)) break;
            if (give_up(tmo, call, t0, polls)) { bad = true; break; }
        }
        const float amax = __uint_as_float(wave_umax(am));
        float inv = __builtin_amdgcn_rcpf(amax * (1.0f / 448.0f));
        if (!(amax > 1e-30f && amax < 1e30f)) inv = amax > 0.f ? 1.0f / (amax / 448.0f) : 0.f;
        *(volatile LAS u64*)(shm + FL_OFF + F_AMAX * 4) = ((u64)(bad ? 2u : 1u) << 32) | __float_as_uint(inv);
        STAMP(5);
    }
    auto quant = [&](const f32x4* v, float inv) {
        u32x4 o;
#pragma unroll
        for (int i = 0; i < 4; ++i) {
            const int w = __builtin_amdgcn_cvt_pk_fp8_f32(v[i][0] * inv, v[i][1] * inv, 0, false);
            o[i] = (unsigned)__builtin_amdgcn_cvt_pk_fp8_f32(v[i][2] * inv, v[i][3] * inv, w, true);
        }
        return o;
    };
    auto wait_scale = [&]() {
        u64 v;
        do v = *(volatile LAS u64*)(shm + FL_OFF + F_AMAX * 4); while (RFL((unsigned)(v >> 32)) == 0u);
        return v;
    };
    if (!remote) {
        const u64 sv = wait_scale();
        u32x4 o = quant(y, __uint_as_float((unsigned)sv));
        if ((unsigned)(sv >> 32) != 1u) o = (u32x4){0x7f7f7f7fu, 0x7f7f7f7fu, 0x7f7f7f7fu, 0x7f7f7f7fu};
        __builtin_amdgcn_raw_buffer_store_b128(o, ro, out0 + wid * 8 * NL, 0, OUT_AUX);
    } else if (wid < 4) {
        const int slot = idx * 8 + SRC_SLOT(grp);
        GAS unsigned* mo = (GAS unsigned*)((GAS unsigned char*)p.ws[grp] + WS_MASK + ((size_t)(call & 1) * GRID + slot) * 2048) + threadIdx.x;
        __hip_atomic_store(mo, *(LAS unsigned*)(cp(wid) + 4096 + lane * 4), RLX_SYS);
        __hip_atomic_store(mo + 256, *(LAS unsigned*)(cp(wid + 4) + 4096 + lane * 4), RLX_SYS);
        f32x4 z[4];
#pragma unroll
        for (int i = 0; i < 4; ++i) {
            y[i] = *(LAS f32x4*)(cp(wid) + lane * 64 + i * 16);
            z[i] = *(LAS f32x4*)(cp(wid + 4) + lane * 64 + i * 16);
        }
        auto rsnd = __builtin_amdgcn_make_buffer_rsrc((void*)(p.ws[grp] + WS_PAY + (size_t)slot * 8192), 0, 8192, 0x00020000);
        int v0 = (unsigned)lane * 16 < fl[F_N + wid] ? (int)threadIdx.x * 16 : 0x10000;
        int v1 = (unsigned)lane * 16 < fl[F_N + wid + 4] ? (int)threadIdx.x * 16 + 4096 : 0x10000;
        asm volatile("" : "+v"(v0), "+v"(v1), "+s"(rsnd), "+v"(y[0]), "+v"(y[1]), "+v"(y[2]), "+v"(y[3]),
                          "+v"(z[0]), "+v"(z[1]), "+v"(z[2]), "+v"(z[3]));
        const u64 sv = wait_scale();
        const float inv = __uint_as_float((unsigned)sv);
        const unsigned orb = tagbits | ((unsigned)(sv >> 32) != 1u ? 0x7f7f7f7fu : 0u);
        u32x4 o0 = quant(y, inv), o1 = quant(z, inv);
#pragma unroll
        for (int i = 0; i < 4; ++i) { o0[i] = (o0[i] & 0x7f7f7f7fu) | orb; o1[i] = (o1[i] & 0x7f7f7f7fu) | orb; }
        __builtin_amdgcn_raw_buffer_store_b128(o0, rsnd, v0, 0, AUX_SYS);
        __builtin_amdgcn_raw_buffer_store_b128(o1, rsnd, v1, 0, AUX_SYS);
        STAMP(6);
    } else {
        const u64 t0 = __builtin_amdgcn_s_memrealtime();
        unsigned polls = 0;
        bool fail = false;
        GAS unsigned* mi = (GAS unsigned*)(myws + WS_MASK + ((size_t)(call & 1) * GRID + b) * 2048) + (threadIdx.x - 256);
        unsigned mw[2];
        for (;;) {
            mw[0] = __hip_atomic_load(mi, RLX_SYS);
            mw[1] = __hip_atomic_load(mi + 256, RLX_SYS);
            if (__all((mw[0] >> 16) == call % 65535u + 1u && (mw[1] >> 16) == call % 65535u + 1u)) break;
            if (give_up(tmo, call, t0, polls)) { fail = true; break; }
        }
        unsigned adr[2][4], sel[2][4];
        int voff[2];
#pragma unroll
        for (int h = 0; h < 2; ++h) {
            const unsigned mask = mw[h] & 0xffffu, c = __builtin_popcount(mask), incl = wave_scan(c);
            voff[h] = (unsigned)lane * 16 < __builtin_amdgcn_readlane(incl, 63) ? ((int)threadIdx.x - 256) * 16 + h * 4096 : 0x10000;
#pragma unroll
            for (int j = 0; j < 4; ++j) {
                adr[h][j] = incl - c + __builtin_popcount(mask & ((1u << (4 * j)) - 1u));
                const unsigned x = spread4((mask >> (4 * j)) & 15u);
                sel[h][j] = ((x * 0x01010100u) & (x * 0xffu)) | ((x ^ 0x01010101u) * 0x0cu);
                asm volatile("" : "+v"(adr[h][j]), "+v"(sel[h][j]));
            }
        }
        auto rrcv = __builtin_amdgcn_make_buffer_rsrc((void*)(p.ws[p.rank & 7] + WS_PAY + (size_t)b * 8192), 0, 8192, 0x00020000);
        LAS char* rb = cp(wid) + 4608;
        u32x4 v[2];
        v[0] = v[1] = (u32x4){0u, 0u, 0u, 0u};
        wait_scale();
        while (!fail) {
            asm volatile("" ::: "memory");
            bool ok = true;
#pragma unroll
            for (int h = 0; h < 2; ++h) {
                v[h] = __builtin_amdgcn_raw_buffer_load_b128(rrcv, voff[h], 0, AUX_SYS);
                const unsigned d = ((v[h][0] ^ tagbits) | (v[h][1] ^ tagbits) | (v[h][2] ^ tagbits) | (v[h][3] ^ tagbits)) & 0x80808080u;
                ok &= voff[h] == 0x10000 || d == 0u;
            }
            if (__all(ok)) break;
            fail = give_up(tmo, call, t0, polls);
        }
        STAMP(7);
#pragma unroll
        for (int h = 0; h < 2; ++h) *(LAS u32x4*)(rb + h * 1040 + lane * 16) = v[h];
        WAIT_LGKM0();
#pragma unroll
        for (int h = 0; h < 2; ++h) {
            u32x4 o;
#pragma unroll
            for (int j = 0; j < 4; ++j) {
                const unsigned a4 = adr[h][j] & ~3u;
                const unsigned lo = *(LAS unsigned*)(rb + h * 1040 + a4), hi = *(LAS unsigned*)(rb + h * 1040 + 4 + a4);
                const unsigned win = __builtin_amdgcn_alignbyte(hi, lo, adr[h][j] & 3u);
                o[j] = (__builtin_amdgcn_perm(0u, win, sel[h][j]) & 0x7f7f7f7fu) | (fail ? 0x7f7f7f7fu : 0u);
            }
            __builtin_amdgcn_raw_buffer_store_b128(o, ro, out0 + (wid - 4 + h * 4) * 8 * NL, 0, OUT_AUX);
        }
#pragma unroll
        for (int h = 0; h < 2; ++h)
            __builtin_amdgcn_raw_buffer_store_b128((u32x4){0u, 0u, 0u, 0u}, rrcv, voff[h], 0, AUX_SYS);
    }
    if (threadIdx.x == 0) __hip_atomic_store(cnt, call, RLX_AGENT);
    STAMP(12);
#endif
}

}

extern "C" void kernel_launch(int rank, int nranks, ncclComm_t comm, hipStream_t stream,
                              void* const* d_in, const int* in_sizes, int n_in,
                              void* d_out, int out_size, const int* in_dtypes, int out_dtype,
                              void* const* d_out_all, void* const* d_ws_all, size_t ws_bytes) {
    (void)comm; (void)in_dtypes; (void)out_dtype; (void)d_out;
    if (nranks != ga::NR || (unsigned)rank >= (unsigned)ga::NR || n_in < 2 || in_sizes[0] != ga::ML * ga::KK ||
        in_sizes[1] != ga::KK * ga::NN || out_size != ga::NR * ga::ML * ga::NL || ws_bytes < ga::WS_END) {
        fprintf(stderr, "kernel_launch: unexpected problem shape\n");
        return;
    }
    HIP_CHECK(hipFuncSetAttribute((const void*)ga::gemm_a2a_kernel, hipFuncAttributeMaxDynamicSharedMemorySize, ga::LDS_BYTES));
    ga::Params p;
    p.x = d_in[0];
    p.w = d_in[1];
    for (int r = 0; r < ga::NR; ++r) {
        p.out[r] = (unsigned char*)d_out_all[r];
        p.ws[r] = (unsigned char*)d_ws_all[r];
    }
    p.rank = rank;
    p.pad = 0;
    hipLaunchKernelGGL(ga::gemm_a2a_kernel, dim3(ga::GRID), dim3(ga::THREADS), ga::LDS_BYTES, stream, p);
    HIP_CHECK(hipGetLastError());
}
```

```cpp
#include <hip/hip_runtime.h>
#include <rccl/rccl.h>
#include <cstdio>

#define HIP_CHECK(expr) do { hipError_t _e = (expr); if (_e != hipSuccess) { \
    fprintf(stderr, "HIP error %s at %s:%d\n", hipGetErrorString(_e), __FILE__, __LINE__); \
    return; } } while (0)

#define NP2_STEPS 12
#define FLAG_SLOT_ 3
#define POLL_SLOT_ 6
#define EARLY_R2 0
#define GPS_ 4
#define XPS_ 8
#define LOADER_PRIO 0
#define LEAD1 3
#define LEAD2 4
#define PRE_MM 4
#define NREP_ 8
#define EARLY_ 0
#define CODE_PF 3
#define RAMP_ 4
#define LOAD_AUX 0

namespace ga {

#define LAS __attribute__((address_space(3)))
#define GAS __attribute__((address_space(1)))
typedef __bf16 bf16x4 __attribute__((ext_vector_type(4)));
typedef __bf16 bf16x8 __attribute__((ext_vector_type(8)));
typedef float f32x4 __attribute__((ext_vector_type(4)));
typedef unsigned u32x4 __attribute__((ext_vector_type(4)));
typedef unsigned long long u64;

constexpr int NR = 8;
constexpr int ML = 512, NN = 4096, KK = 4096, NL = 512;
constexpr int BK = 64, NKS = KK / BK;
constexpr int NP2 = NP2_STEPS;
constexpr int NP1 = (NKS - NP2) / 2;
constexpr int NS = NP1 + NP2;
static_assert(2 * NP1 + NP2 == NKS && NP2 >= 6, "K split");
constexpr int ST = 4, STAGE_B = 32768, W_OFF = 16384;
constexpr int KB_OFF = ST * STAGE_B;
constexpr int FL_OFF = KB_OFF + 16384;
constexpr int LDS_BYTES = FL_OFF + 512;
constexpr int F_WMAX = 8, F_AMAX = 16, F_CALL = 18, F_R2 = 19;
constexpr int GRID = 256, THREADS = 512, RAMP = RAMP_;
static_assert(RAMP + 4 <= NP1, "prefetch ramp");
constexpr int FLAG_SLOT = FLAG_SLOT_, POLL_SLOT = POLL_SLOT_;
constexpr int NXB = 16, XPS = XPS_, GPS = GPS_;
static_assert(NXB % XPS == 0 && 8 % GPS == 0 && FLAG_SLOT >= 8 / GPS && POLL_SLOT >= FLAG_SLOT && POLL_SLOT + 3 + NXB / XPS <= NP2, "hand-off schedule");

constexpr size_t WS_CNT = 0x0, WS_TMO = 0x1000, WS_LMAX = 0x2000, WS_GMAX = 0x4000  ,
                 WS_XFLAG = 0x8000, WS_STAMP = 0x10000, WS_XCH = 0x100000,
                 WS_MASK = WS_XCH + (size_t)GRID * 32768,
                 WS_PAY = WS_MASK + 2 * (size_t)GRID * 2048,
                 WS_EARLY = WS_PAY + (size_t)GRID * 8192,
                 WS_END = WS_EARLY + 2 * (size_t)GRID * 4096;
constexpr int NREP = NREP_;
constexpr int EARLY = EARLY_;
static_assert(EARLY % 16 == 0 && EARLY <= 512, "early part");
constexpr u64 DEADLINE_TICKS = 50000000ull;

struct Params {
    const void* x;
    const void* w;
    unsigned char* myws;
    unsigned char* myout;
    int rank;
    int pad;
    unsigned char* ws[NR];
};

#define RLX_AGENT __ATOMIC_RELAXED, __HIP_MEMORY_SCOPE_AGENT
#define RLX_SYS __ATOMIC_RELAXED, __HIP_MEMORY_SCOPE_SYSTEM
#define BARRIER() asm volatile("s_barrier" ::: "memory")
#define WAIT_VM(n) asm volatile("s_waitcnt vmcnt(" #n ")" ::: "memory")
#define WAIT_LGKM0() asm volatile("s_waitcnt lgkmcnt(0)" ::: "memory")
#define RFL(x) __builtin_amdgcn_readfirstlane(x)

#define SRC_SLOT(d) (p.rank & 7)
#define OUT_PTR p.myout
#define OUT_ROW0 (grp * ML)
#define STAMP(i) do {} while (0)

#if defined(__HIP_DEVICE_COMPILE__)
constexpr int AUX_SYS = 17;
__device__ __forceinline__ bool give_up(GAS unsigned* tmo, unsigned call, u64 t0, unsigned& polls) {
    if ((++polls & 31u) != 0u) return false;
    if (RFL(__hip_atomic_load(tmo, RLX_AGENT)) == call || __builtin_amdgcn_s_memrealtime() - t0 > DEADLINE_TICKS) {
        __hip_atomic_store(tmo, call, RLX_AGENT);
        return true;
    }
    return false;
}
typedef float f32x2 __attribute__((ext_vector_type(2)));
typedef __bf16 bf16x2 __attribute__((ext_vector_type(2)));
__device__ __forceinline__ unsigned pack2(float a, float b) {
    return __builtin_bit_cast(unsigned, __builtin_convertvector((f32x2){a, b}, bf16x2));
}
__device__ __forceinline__ u32x4 pack_bf16(f32x4 a, f32x4 b) {
    return (u32x4){pack2(a[0], a[1]), pack2(a[2], a[3]), pack2(b[0], b[1]), pack2(b[2], b[3])};
}
__device__ __forceinline__ unsigned umax(unsigned a, unsigned b) { return a > b ? a : b; }
__device__ __forceinline__ unsigned umin(unsigned a, unsigned b) { return a < b ? a : b; }
__device__ __forceinline__ unsigned wave_scan(unsigned v) {
    v += (unsigned)__builtin_amdgcn_update_dpp(0, (int)v, 0x111, 0xF, 0xF, true);
    v += (unsigned)__builtin_amdgcn_update_dpp(0, (int)v, 0x112, 0xF, 0xF, true);
    v += (unsigned)__builtin_amdgcn_update_dpp(0, (int)v, 0x114, 0xF, 0xF, true);
    v += (unsigned)__builtin_amdgcn_update_dpp(0, (int)v, 0x118, 0xF, 0xF, true);
    v += (unsigned)__builtin_amdgcn_update_dpp(0, (int)v, 0x142, 0xA, 0xF, false);
    v += (unsigned)__builtin_amdgcn_update_dpp(0, (int)v, 0x143, 0xC, 0xF, false);
    return v;
}
__device__ __forceinline__ unsigned spread4(unsigned nib) { return (nib * 0x00204081u) & 0x01010101u; }
__device__ __forceinline__ unsigned wave_umax(unsigned v) {
    v = umax(v, (unsigned)__builtin_amdgcn_update_dpp(0, (int)v, 0xB1, 0xF, 0xF, true));
    v = umax(v, (unsigned)__builtin_amdgcn_update_dpp(0, (int)v, 0x4E, 0xF, 0xF, true));
    v = umax(v, (unsigned)__builtin_amdgcn_update_dpp(0, (int)v, 0x141, 0xF, 0xF, true));
    v = umax(v, (unsigned)__builtin_amdgcn_update_dpp(0, (int)v, 0x140, 0xF, 0xF, true));
    return umax(umax(__builtin_amdgcn_readlane(v, 0), __builtin_amdgcn_readlane(v, 16)),
                umax(__builtin_amdgcn_readlane(v, 32), __builtin_amdgcn_readlane(v, 48)));
}
#endif

__global__ __launch_bounds__(THREADS, 2) void gemm_a2a_kernel(Params p) {
#if defined(__HIP_DEVICE_COMPILE__)
    extern __shared__ __attribute__((aligned(16))) char shm_raw[];
    LAS char* shm = (LAS char*)shm_raw;
    typedef u64 u64x2 __attribute__((ext_vector_type(2)));
    u64x2 xw;
    asm volatile("s_load_dwordx4 %0, %1, 0x0" : "=s"(xw) : "s"(__builtin_amdgcn_kernarg_segment_ptr()));
    __builtin_amdgcn_sched_barrier(0);
    const int b = blockIdx.x;
    const int grp = b & 7, idx = b >> 3;
    const int tn = idx & 3, tm = (idx >> 2) & 3, hb = idx >> 4;
    const int wid = RFL(threadIdx.x >> 6);
    const int lane = threadIdx.x & 63;
    auto role_rsrc = [&](int lw) {
        return __builtin_amdgcn_make_buffer_rsrc((void*)((lw < 2 ? xw[1] : xw[0]) - 4096), 0,
                                                 (lw < 2 ? KK * NN * 2 : ML * KK * 2) + 4096, 0x00020000);
    };
    auto role_off = [&](int lw, int i) {
        const int mw = (lw - 2) >> 31;
        const int o = (mw & ((grp * NL + tn * 128 + (lw & 1) * 64) * 2)) | (~mw & ((tm * 128 + (lw & 1) * 64) * (KK * 2)));
        const int sw = lw < 2 ? ((lane >> 3) & 2) : (lane >> 4);
        return (lane >> 3) * (KK * 2) + (((lane & 7) ^ sw ^ ((i & 1) << 2)) << 4) + o + i * (8 * KK * 2) - (i & 3) * 1024 + 4096;
    };
    static_assert(KK == NN, "x and W rows have the same pitch");
    {
        const int lw = wid >> 1, g = wid & 1;
        int so = (hb * NP1 * (BK * 2)) << (lw < 2 ? 12 : 0);
        LAS char* dst = shm + (lw < 2 ? W_OFF : 0) + (lw & 1) * 8192 + g * 4096;
        int e = role_off(lw, 0) + g * (32 * KK * 2), o = role_off(lw, 1) + g * (32 * KK * 2);
        asm volatile("s_waitcnt lgkmcnt(0)" : "+s"(xw), "+v"(e), "+v"(o), "+s"(so), "+s"(dst));
        const auto rs0 = role_rsrc(lw);
        __builtin_amdgcn_raw_ptr_buffer_load_lds(rs0, dst, 16, e, so, 0, LOAD_AUX);
        __builtin_amdgcn_raw_ptr_buffer_load_lds(rs0, dst, 16, o, so, 1024, LOAD_AUX);
        __builtin_amdgcn_raw_ptr_buffer_load_lds(rs0, dst, 16, e + 16 * KK * 2 - 2048, so, 2048, LOAD_AUX);
        __builtin_amdgcn_raw_ptr_buffer_load_lds(rs0, dst, 16, o + 16 * KK * 2 - 2048, so, 3072, LOAD_AUX);
        __builtin_amdgcn_sched_barrier(0);
        asm volatile("" ::: "memory");
    }
    auto code_prefetch = [&]() {
        u64 pc;
        int len;
        asm volatile("s_getpc_b64 %0\n.Lga_here%=:\n\ts_mov_b32 %1, .Lga_end_a-.Lga_here%=\n\t"
                     "s_max_i32 %1, %1, .Lga_end_b-.Lga_here%=\n\ts_max_i32 %1, %1, .Lga_end_c-.Lga_here%="
                     : "=s"(pc), "=s"(len) :: "scc");
        auto rc = __builtin_amdgcn_make_buffer_rsrc((void*)pc, 0, len, 0x00020000);
        __builtin_amdgcn_raw_ptr_buffer_load_lds(rc, shm + FL_OFF + 256, 4, lane < 8 ? (idx * 8 + lane) * 128 : 0x7fffff00, 0, 0, 0);
    };
    if (wid == 0) code_prefetch();
    GAS unsigned char* myws = (GAS unsigned char*)p.myws;
    GAS unsigned* tmo = (GAS unsigned*)(myws + WS_TMO);
    GAS unsigned* cnt = (GAS unsigned*)(myws + WS_CNT) + b;
    STAMP(0);

    volatile LAS unsigned* fl = (volatile LAS unsigned*)(shm + FL_OFF);
    auto x2_off = [&](int i) { return (i >> 3) * STAGE_B + 8192 + (i & 7) * 1024; };
    auto r3_off = [&](int c, int k) { return c * STAGE_B + W_OFF + k * 1024; };

    f32x4 fin[8];
    const int wr = (wid >> 1) & 1, wc = wid & 1;
    unsigned char* pw = nullptr;

    if (wid >= 4) {
        if (LOADER_PRIO) __builtin_amdgcn_s_setprio(LOADER_PRIO);
        const int lw = wid - 4;
        const bool isw = lw < 2;
        const auto rs = role_rsrc(lw);
        int vo[8];
#pragma unroll
        for (int i = 0; i < 8; ++i) vo[i] = role_off(lw, i);
        const int lds0 = (isw ? W_OFF : 0) + (lw & 1) * 8192, shl = isw ? 12 : 0;
        auto issue4 = [&](int s, int so, int g) {
            LAS char* dst = shm + (s & (ST - 1)) * STAGE_B + lds0 + g * 4096;
            __builtin_amdgcn_raw_ptr_buffer_load_lds(rs, dst, 16, vo[g * 4 + 0], so, 0, LOAD_AUX);
            __builtin_amdgcn_raw_ptr_buffer_load_lds(rs, dst, 16, vo[g * 4 + 1], so, 1024, LOAD_AUX);
            __builtin_amdgcn_raw_ptr_buffer_load_lds(rs, dst, 16, vo[g * 4 + 2], so, 2048, LOAD_AUX);
            __builtin_amdgcn_raw_ptr_buffer_load_lds(rs, dst, 16, vo[g * 4 + 3], so, 3072, LOAD_AUX);
        };
        unsigned c0 = 0;
        if (lw == 3) {
            c0 = __hip_atomic_load(cnt, RLX_AGENT);
            pw = p.ws[lane >> 3];
        }
        int ord;
        {
            const bool p2 = lane >= NP1;
            const int t = p2 ? lane - NP1 : lane, nt = p2 ? NP2 : NP1, lead = p2 ? LEAD2 : LEAD1, sh = p2 ? 3 : 2;
            const int q = p2 ? ((tm * 2 + hb + 2 * tn) & 7) : ((tm + tn) & 3);
            auto cls_upto = [&](int m) { return m >= q ? ((m - q) >> sh) + 1 : 0; };
            auto led_upto = [&](int m) { return m > lead ? cls_upto(m) - cls_upto(lead) : 0; };
            const bool led = (t & ((1 << sh) - 1)) == q && t > lead;
            const int m = t - lead - 1, hi = t + lead < nt - 1 ? t + lead : nt - 1;
            int pos = led ? led_upto(t) - 1 + m + 1 - led_upto(m) : t - led_upto(t) + led_upto(hi);
            pos = lane < NS ? pos + (p2 ? NP1 : 0) : lane;
            const int so = (((t + (p2 ? 2 * NP1 : hb * NP1)) * (BK * 2)) << shl) + ((p2 && !isw) ? hb * 64 * (KK * 2) : 0);
            ord = __builtin_amdgcn_ds_permute(pos << 2, so);
        }
#define VMW(n) asm volatile("s_waitcnt vmcnt(%0)" :: "i"(n) : "memory")
        auto issue8 = [&](int s) {
            const int so = __builtin_amdgcn_readlane(ord, s);
            issue4(s, so, 0);
            issue4(s, so, 1);
        };
        issue8(1);
        VMW(8);
        for (int s = 0; s < RAMP; ++s) {
            BARRIER();
            issue8(s + 2);
            VMW(8);
        }
        const int nst = (lw == 3) ? NP1 : NS;
        BARRIER();
        issue8(RAMP + 2);
        issue8(RAMP + 3);
        VMW(16);
        for (int s = RAMP + 1; s < nst - 3; ++s) {
            BARRIER();
            issue8(s + 3);
            VMW(16);
        }
        BARRIER();
        VMW(8);
        BARRIER();
        VMW(0);
        if (lw != 3) {
            BARRIER();
        } else {
            __builtin_amdgcn_s_setprio(0);
            const unsigned call = RFL(c0) + 1;
            fl[F_CALL] = call;
            fl[F_AMAX + 1] = 0;
            fl[F_R2] = 0;
            asm volatile("" : "+v"(pw));
            GAS u64* xf = (GAS u64*)(myws + WS_XFLAG) + (b ^ 128);
            auto rp = __builtin_amdgcn_make_buffer_rsrc((void*)(p.myws + WS_XCH + (size_t)(b ^ 128) * 32768), 0, 32768, 0x00020000);
            auto fetch = [&](int i0, int i1) {
#pragma unroll
                for (int i = i0; i < i1; ++i)
                    __builtin_amdgcn_raw_ptr_buffer_load_lds(rp, shm + x2_off(i), 16, lane * 16, i * 1024, 0, AUX_SYS);
            };
            code_prefetch();
            WAIT_LGKM0();
            BARRIER();
            constexpr int NXS = NXB / XPS;
            for (int u = 0; u <= POLL_SLOT; ++u) BARRIER();
            auto both = [&](u64 v) { return RFL((int)((unsigned)v >= call && (unsigned)(v >> 32) >= call)) != 0; };
            u64 f;
            asm volatile("global_load_dwordx2 %0, %1, off sc0 sc1\n\ts_barrier\n\ts_barrier\n\ts_waitcnt vmcnt(0)"
                         : "=&v"(f) : "v"(xf) : "memory");
            const bool got = both(f);
            if (got) STAMP(8);
#pragma unroll
            for (int k = 0; k < NXS; ++k) {
                if (k > 0) BARRIER();
                if (got) fetch(k * XPS, (k + 1) * XPS);
            }
            for (int u = POLL_SLOT + 2 + NXS; u < NP2 - 1; ++u) BARRIER();
            if (got && EARLY_R2) {
                WAIT_VM(0);
                fl[F_R2] = 1;
                WAIT_LGKM0();
                STAMP(9);
            }
            BARRIER();
            if (!got || !EARLY_R2) {
                bool ok = true;
                if (!got) {
                    const u64 t0 = __builtin_amdgcn_s_memrealtime();
                    unsigned polls = 0;
                    while (!both(__hip_atomic_load(xf, RLX_SYS)))
                        if (give_up(tmo, call, t0, polls)) { ok = false; break; }
                    STAMP(8);
                    if (ok) fetch(0, NXB);
                }
                WAIT_VM(0);
                fl[F_R2] = ok ? 1 : 2;
                WAIT_LGKM0();
                STAMP(9);
            }
        }
    } else {
        const int gq = lane >> 4, qq = (lane & 15) >> 2, pp = lane & 3;
        LAS char* bx[2][2][2];
        LAS char* bw[2][4];
#pragma unroll
        for (int hi = 0; hi < 2; ++hi) {
#pragma unroll
            for (int ks = 0; ks < 2; ++ks) {
                const int xo = hi * 65536 + (lane & 15) * 128 + (((4 * ks + gq) ^ ((lane >> 1) & 7)) << 4);
                bx[hi][ks][0] = shm + xo + wr * 12288;
                bx[hi][ks][1] = shm + xo + 4096 + wr * 4096;
                asm volatile("" : "+v"(bx[hi][ks][0]), "+v"(bx[hi][ks][1]));
            }
#pragma unroll
            for (int j = 0; j < 4; ++j) {
                bw[hi][j] = shm + hi * 65536 + W_OFF + wc * 8192 + 128 * (8 * gq + qq) + ((j ^ ((qq >> 1) | ((gq & 1) << 1))) << 5) + 8 * pp;
                asm volatile("" : "+v"(bw[hi][j]));
            }
        }
        f32x4 acc[4][4];
#pragma unroll
        for (int k = 0; k < 16; ++k) acc[k >> 2][k & 3] = (f32x4){0.f, 0.f, 0.f, 0.f};
        struct Frag { bf16x8 x[4], w[4]; };
        auto rd = [&](Frag& f, int sl, int ks, int ni) {
#pragma unroll
            for (int j = 0; j < 4; ++j) {
                if (j < ni) f.x[j] = *(LAS bf16x8*)(bx[sl >> 1][ks][j >> 1] + (sl & 1) * STAGE_B + (j & 1) * 2048);
                LAS char* wb = bw[sl >> 1][j] + (sl & 1) * STAGE_B + ks * 4096;
                const bf16x4 lo = __builtin_amdgcn_ds_read_tr16_b64_v4bf16((LAS bf16x4*)wb);
                const bf16x4 hi = __builtin_amdgcn_ds_read_tr16_b64_v4bf16((LAS bf16x4*)(wb + 512));
                f.w[j] = __builtin_shufflevector(lo, hi, 0, 1, 2, 3, 4, 5, 6, 7);
            }
        };
        auto mm = [&](const Frag& f, int n0, int n1) {
#pragma unroll
            for (int n = n0; n < n1; ++n)
                acc[n >> 2][n & 3] = __builtin_amdgcn_mfma_f32_16x16x32_bf16(f.w[n & 3], f.x[n >> 2], acc[n >> 2][n & 3], 0, 0, 0);
        };
#define SGB(m, n) __builtin_amdgcn_sched_group_barrier(m, n, 0)
#define SCHED_FENCE() __builtin_amdgcn_sched_barrier(0)
#define GROUP1_END() do { _Pragma("unroll") for (int _i = 0; _i < 12; ++_i) { SGB(0x8, 1); SGB(0x100, 1); } SGB(0x8, 4); SCHED_FENCE(); } while (0)
#define GROUP2_END() do { _Pragma("unroll") for (int _i = 0; _i < 5; ++_i) { SGB(0x8, 1); SGB(0x100, 2); } SGB(0x8, 3); SCHED_FENCE(); } while (0)
        Frag fa, fb;
        auto stage_barrier = [&](int s, int npre) {
            __builtin_amdgcn_s_waitcnt(0xC07F);
            SCHED_FENCE();
            if (npre) mm(fb, 0, npre);
            SCHED_FENCE();
            BARRIER();
            SCHED_FENCE();
        };
        WAIT_VM(0);
        stage_barrier(0, 0);
        STAMP(11);
        rd(fa, 0, 0, 4);
        SCHED_FENCE();
        rd(fb, 0, 1, 4); mm(fa, 0, 16); GROUP1_END();
        auto stage1 = [&](int s, int sl) {
            stage_barrier(s, PRE_MM);
            rd(fa, sl, 0, 4); mm(fb, PRE_MM, 16); GROUP1_END();
            rd(fb, sl, 1, 4); mm(fa, 0, 16); GROUP1_END();
        };
        for (int s = 1; s + 3 < NP1; s += 4) { stage1(s, 1); stage1(s + 1, 2); stage1(s + 2, 3); stage1(s + 3, 0); }
#pragma unroll
        for (int s = NP1 - (NP1 - 1) % 4; s < NP1; ++s) stage1(s, s & 3);
#pragma unroll
        for (int k = 0; k < 4; ++k) bx[k >> 1][k & 1][0] -= wr * 8192;
        stage_barrier(NP1, PRE_MM);
        rd(fa, NP1 & 3, 0, 2);
        mm(fb, PRE_MM, 16);
        SCHED_FENCE();
        STAMP(1);
        const bool giver = (wr != hb);
        u32x4 pk[8];
        if (giver) {
#pragma unroll
            for (int k = 0; k < 8; ++k) pk[k] = pack_bf16(acc[k >> 1][(k & 1) * 2], acc[k >> 1][(k & 1) * 2 + 1]);
#pragma unroll
            for (int k = 0; k < 8; ++k) acc[k >> 2][k & 3] = (f32x4){0.f, 0.f, 0.f, 0.f};
        }
        auto rdump = __builtin_amdgcn_make_buffer_rsrc((void*)(p.myws + WS_XCH + (size_t)b * 32768), 0, 32768, 0x00020000);
        SCHED_FENCE();
        auto add_r2 = [&]() {
#pragma unroll
            for (int k = 0; k < 4; ++k) {
                const u32x4 v = *(LAS u32x4*)(shm + x2_off(wc * 8 + wr * 4) + k * 1024 + lane * 16);
#pragma unroll
                for (int e = 0; e < 4; ++e) {
                    acc[k >> 1][(k & 1) * 2 + (e >> 1)][(e & 1) * 2] += __uint_as_float(v[e] << 16);
                    acc[k >> 1][(k & 1) * 2 + (e >> 1)][(e & 1) * 2 + 1] += __uint_as_float(v[e] & 0xffff0000u);
                }
            }
        };
        bool r2_in = false;
#pragma unroll
        for (int u = 0; u < NP2; ++u) {
            if (u > 0) { stage_barrier(NP1 + u, 0); rd(fa, (NP1 + u) & 3, 0, 2); mm(fb, 0, 8); GROUP2_END(); }
            if (u == NP2 - 1) {
                r2_in = RFL(fl[F_R2]) != 0u;
                asm volatile("" ::: "memory");
                if (r2_in) add_r2();
                SCHED_FENCE();
            }
            rd(fb, (NP1 + u) & 3, 1, 2); mm(fa, 0, 8); GROUP2_END();
            if (u * GPS < 8) {
                if (giver) {
#pragma unroll
                    for (int k = u * GPS; k < (u + 1) * GPS; ++k)
                        __builtin_amdgcn_raw_buffer_store_b128(pk[k], rdump, lane * 16, (wc * 8 + (((k >> 2) ^ wr) << 2) + (k & 3)) * 1024, AUX_SYS);
                    asm volatile("s_nop 1" ::: "memory");
                } else {
#pragma unroll
                    for (int k = u * GPS; k < (u + 1) * GPS; ++k)
                        *(LAS f32x4*)(shm + KB_OFF + wc * 8192 + k * 1024 + lane * 16) = acc[2 + (k >> 2)][k & 3];
                }
                SCHED_FENCE();
            }
            if (u == FLAG_SLOT && giver) {
                WAIT_VM(0);
                if (lane == 0) __hip_atomic_store((GAS unsigned*)(myws + WS_XFLAG) + b * 2 + wc, RFL(fl[F_CALL]), RLX_SYS);
                STAMP(10);
                SCHED_FENCE();
            }
            if (u == 8 / GPS + 1 && giver) {
#pragma unroll
                for (int k = 0; k < 8; ++k) acc[k >> 2][k & 3] += *(LAS f32x4*)(shm + KB_OFF + wc * 8192 + k * 1024 + lane * 16);
                SCHED_FENCE();
            }
        }
        mm(fb, 0, 8);
        if (!r2_in) {
            while (RFL(fl[F_R2]) == 0u) {}
            asm volatile("" ::: "memory");
            add_r2();
        }
        STAMP(2);
        float mx = 0.f;
#pragma unroll
        for (int k = 0; k < 8; ++k) {
            fin[k] = acc[k >> 2][k & 3];
            mx = fmaxf(fmaxf(mx, fin[k][0]), fin[k][1]);
            mx = fmaxf(fmaxf(mx, fin[k][2]), fin[k][3]);
        }
        fl[F_WMAX + wid] = wave_umax(__float_as_uint(mx) & 0x7fffffffu);
        WAIT_LGKM0();
    }
    BARRIER();
    STAMP(3);
    const unsigned call = RFL(fl[F_CALL]);
    if (wid < 4) {
#pragma unroll
        for (int k = 0; k < 8; ++k) {
#pragma unroll
            for (int e = 0; e < 4; ++e) fin[k][e] = fmaxf(fin[k][e], 0.f);
            *(LAS f32x4*)(shm + r3_off(wc, wr * 8 + k) + ((lane ^ ((k & 3) << 2)) << 4)) = fin[k];
        }
        WAIT_LGKM0();
    } else if (wid == 7) {
        const u32x4 m4 = *(volatile LAS u32x4*)(shm + FL_OFF + F_WMAX * 4);
        const u64 gran = ((u64)call << 32) | umax(umax(m4[0], m4[1]), umax(m4[2], m4[3]));
        if (lane == 0) __hip_atomic_store((GAS u64*)(myws + WS_LMAX) + b, gran, RLX_AGENT);
    }
    BARRIER();
    const bool remote = grp != (p.rank & 7);
    const unsigned tagbits = spread4(call % 15u + 1u) << 7;
    const unsigned tag16 = call % 65535u + 1u;
    const unsigned tag2 = (((call % 3u + 1u) & 1u) << 15) | (((call % 3u + 1u) >> 1) << 31);
    LAS char* cp = shm + 2 * STAGE_B + (wid < 4 ? wid * 12288 : 49152 + (wid - 4) * 4096);
    auto ro = __builtin_amdgcn_make_buffer_rsrc((void*)OUT_PTR, 0, NR * ML * NL, 0x00020000);
    const int out0 = (OUT_ROW0 + tm * 128 + hb * 64 + (wid & 3) * 16 + (lane >> 3)) * NL + tn * 128 + (lane & 7) * 16;
    auto publish_scale = [&](unsigned am, bool bad) {
        const float amax = __uint_as_float(am);
        float inv = __builtin_amdgcn_rcpf(amax * (1.0f / 448.0f));
        if (!(amax > 1e-30f && amax < 1e30f)) inv = amax > 0.f ? 1.0f / (amax / 448.0f) : 0.f;
        *(volatile LAS u64*)(shm + FL_OFF + F_AMAX * 4) = ((u64)(bad ? 2u : 1u) << 32) | __float_as_uint(inv);
    };
    if (wid == 7) {
        const u64 t0 = __builtin_amdgcn_s_memrealtime();
        unsigned polls = 0;
        const size_t goff = WS_GMAX + (size_t)(call & 1) * (NREP * 128);
        if (b == (p.rank & 7)) {
            GAS u64* lmax = (GAS u64*)(myws + WS_LMAX);
            unsigned gm;
            bool bad = false;
            for (;;) {
                bool ok = true;
                gm = 0;
#pragma unroll
                for (int k = 0; k < 4; ++k) {
                    const u64 v = __hip_atomic_load(lmax + k * 64 + lane, RLX_AGENT);
                    ok &= (unsigned)(v >> 32) >= call;
                    gm = umax(gm, (unsigned)v);
                }
                if (__all(ok)) break;
                if (give_up(tmo, call, t0, polls)) { bad = true; break; }
            }
            gm = wave_umax(gm);
            if (!bad) {
#pragma unroll
                for (int k = 0; k < NREP / 8; ++k)
                    __hip_atomic_store((GAS u64*)(pw + goff + (k * 8 + (lane & 7)) * 128) + SRC_SLOT(lane >> 3), ((u64)call << 32) | gm, RLX_SYS);
            }
            STAMP(4);
        }
        GAS u64* gp = (GAS u64*)(myws + goff + (b & (NREP - 1)) * 128) + (lane & 7);
        unsigned am;
        bool bad = fl[F_R2] != 1u;
        for (;;) {
            const u64 v = __hip_atomic_load(gp, RLX_SYS);
            am = (unsigned)v;
            if (__all((unsigned)(v >> 32) >= call)) break;
            if (give_up(tmo, call, t0, polls)) { bad = true; break; }
        }
        publish_scale(wave_umax(am), bad);
        STAMP(5);
    }
    auto quant = [&](const f32x4* v, float inv) {
        u32x4 o;
#pragma unroll
        for (int i = 0; i < 4; ++i) {
            const int w = __builtin_amdgcn_cvt_pk_fp8_f32(v[i][0] * inv, v[i][1] * inv, 0, false);
            o[i] = (unsigned)__builtin_amdgcn_cvt_pk_fp8_f32(v[i][2] * inv, v[i][3] * inv, w, true);
        }
        return o;
    };
    auto wait_scale = [&]() {
        u64 v;
        do v = *(volatile LAS u64*)(shm + FL_OFF + F_AMAX * 4); while (RFL((unsigned)(v >> 32)) == 0u);
        return v;
    };
    if (wid < 4) {
        f32x4 y[2][4];
#pragma unroll
        for (int h = 0; h < 2; ++h)
#pragma unroll
            for (int i = 0; i < 4; ++i)
                y[h][i] = *(LAS f32x4*)(shm + r3_off((lane >> 2) & 1, wid * 4 + (lane & 3)) +
                                        (((h * 8 + (lane >> 3) + 16 * i) ^ ((lane & 3) << 2)) << 4));
        int v0 = out0, v1 = out0 + 8 * NL;
        unsigned andb = 0xffffffffu, orb = 0u;
        auto rsnd = ro;
        if (remote) {
            unsigned mask[2] = {0u, 0u};
#pragma unroll
            for (int e = 0; e < 32; ++e) mask[e >> 4] |= (y[e >> 4][(e >> 2) & 3][e & 3] > 0.f ? 1u : 0u) << (e & 15);
            const int slot = idx * 8 + SRC_SLOT(grp);
            GAS unsigned* mo = (GAS unsigned*)((GAS unsigned char*)p.ws[grp] + WS_MASK + ((size_t)(call & 1) * GRID + slot) * 2048) + threadIdx.x;
            __hip_atomic_store(mo, mask[0] | (tag16 << 16), RLX_SYS);
            __hip_atomic_store(mo + 256, mask[1] | (tag16 << 16), RLX_SYS);
            const unsigned c = __builtin_popcount(mask[0]) | (__builtin_popcount(mask[1]) << 16), incl = wave_scan(c);
            const unsigned tot = __builtin_amdgcn_readlane(incl, 63), n = (tot & 0xffffu) + (tot >> 16);
#pragma unroll
            for (int h = 0; h < 2; ++h) {
                unsigned pos = (h ? (tot & 0xffffu) + ((incl - c) >> 16) : ((incl - c) & 0xffffu)) * 4;
#pragma unroll
                for (int e = 0; e < 16; ++e)
                    if (y[h][e >> 2][e & 3] > 0.f) { *(LAS float*)(cp + pos) = y[h][e >> 2][e & 3]; pos += 4; }
            }
            WAIT_LGKM0();
            if (EARLY) {
                auto re = __builtin_amdgcn_make_buffer_rsrc((void*)(p.ws[grp] + WS_EARLY + ((size_t)(call & 1) * GRID + slot) * 4096), 0, 4096, 0x00020000);
                const u32x4 e8 = pack_bf16(*(LAS f32x4*)(cp + lane * 32), *(LAS f32x4*)(cp + lane * 32 + 16));
                __builtin_amdgcn_raw_buffer_store_b128((e8 & 0x7fff7fffu) | tag2, re, lane * 8 < EARLY ? wid * 1024 + lane * 16 : 0x10000, 0, AUX_SYS);
            }
#pragma unroll
            for (int h = 0; h < 2; ++h)
#pragma unroll
                for (int i = 0; i < 4; ++i) y[h][i] = *(LAS f32x4*)(cp + EARLY * 4 + h * 4096 + lane * 64 + i * 16);
            rsnd = __builtin_amdgcn_make_buffer_rsrc((void*)(p.ws[grp] + WS_PAY + (size_t)slot * 8192), 0, 8192, 0x00020000);
            v0 = (unsigned)lane * 16 + EARLY < n ? wid * 2048 + lane * 16 : 0x10000;
            v1 = (unsigned)lane * 16 + 1024 + EARLY < n ? wid * 2048 + 1024 + lane * 16 : 0x10000;
            andb = 0x7f7f7f7fu;
            orb = tagbits;
        }
        asm volatile("" : "+v"(v0), "+v"(v1), "+s"(rsnd), "+s"(andb), "+s"(orb), "+v"(y[0][0]), "+v"(y[0][1]), "+v"(y[0][2]), "+v"(y[0][3]),
                          "+v"(y[1][0]), "+v"(y[1][1]), "+v"(y[1][2]), "+v"(y[1][3]));
        const u64 sv = wait_scale();
        const float inv = __uint_as_float((unsigned)sv);
        if ((unsigned)(sv >> 32) != 1u) orb |= 0x7f7f7f7fu;
        u32x4 o = quant(y[0], inv);
        __builtin_amdgcn_raw_buffer_store_b128((o & andb) | orb, rsnd, v0, 0, AUX_SYS);
        o = quant(y[1], inv);
        __builtin_amdgcn_raw_buffer_store_b128((o & andb) | orb, rsnd, v1, 0, AUX_SYS);
        STAMP(6);
        asm volatile(".Lga_end_a:");
    } else if (remote) {
        const u64 t0 = __builtin_amdgcn_s_memrealtime();
        unsigned polls = 0;
        bool fail = false;
        GAS unsigned* mi = (GAS unsigned*)(myws + WS_MASK + ((size_t)(call & 1) * GRID + b) * 2048) + (threadIdx.x - 256);
        unsigned mw[2];
        for (;;) {
            mw[0] = __hip_atomic_load(mi, RLX_SYS);
            mw[1] = __hip_atomic_load(mi + 256, RLX_SYS);
            if (__all((mw[0] >> 16) == tag16 && (mw[1] >> 16) == tag16)) break;
            if (give_up(tmo, call, t0, polls)) { fail = true; break; }
        }
        const unsigned c = __builtin_popcount(mw[0] & 0xffffu) | (__builtin_popcount(mw[1] & 0xffffu) << 16), incl = wave_scan(c);
        const unsigned tot = __builtin_amdgcn_readlane(incl, 63), n = (tot & 0xffffu) + (tot >> 16);
        unsigned adr[2][4], sel[2][4];
        int voff[2];
#pragma unroll
        for (int h = 0; h < 2; ++h) {
            const unsigned mask = mw[h] & 0xffffu, p0 = h ? (tot & 0xffffu) + ((incl - c) >> 16) : ((incl - c) & 0xffffu);
            voff[h] = (unsigned)lane * 16 + h * 1024 + EARLY < n ? (wid - 4) * 2048 + h * 1024 + lane * 16 : 0x10000;
#pragma unroll
            for (int j = 0; j < 4; ++j) {
                adr[h][j] = p0 + __builtin_popcount(mask & ((1u << (4 * j)) - 1u));
                const unsigned x = spread4((mask >> (4 * j)) & 15u);
                sel[h][j] = ((x * 0x01010100u) & (x * 0xffu)) | ((x ^ 0x01010101u) * 0x0cu);
                asm volatile("" : "+v"(adr[h][j]), "+v"(sel[h][j]));
            }
        }
        auto rrcv = __builtin_amdgcn_make_buffer_rsrc((void*)(p.myws + WS_PAY + (size_t)b * 8192), 0, 8192, 0x00020000);
        u32x4 v[2];
        v[0] = v[1] = (u32x4){0u, 0u, 0u, 0u};
        if (EARLY) {
            auto re = __builtin_amdgcn_make_buffer_rsrc((void*)(p.myws + WS_EARLY + ((size_t)(call & 1) * GRID + b) * 4096), 0, 4096, 0x00020000);
            const bool mine = lane * 8 < EARLY;
            u32x4 e8 = {0u, 0u, 0u, 0u};
            while (!fail) {
                asm volatile("" ::: "memory");
                e8 = __builtin_amdgcn_raw_buffer_load_b128(re, mine ? (wid - 4) * 1024 + lane * 16 : 0x10000, 0, AUX_SYS);
                if (__all(!mine || (((e8[0] ^ tag2) | (e8[1] ^ tag2) | (e8[2] ^ tag2) | (e8[3] ^ tag2)) & 0x80008000u) == 0u)) break;
                fail = give_up(tmo, call, t0, polls);
            }
            const u64 sv = wait_scale();
            fail |= (unsigned)(sv >> 32) != 1u;
            const float inv = __uint_as_float((unsigned)sv);
            unsigned q[2];
#pragma unroll
            for (int i = 0; i < 2; ++i) {
                const int w = __builtin_amdgcn_cvt_pk_fp8_f32(__uint_as_float((e8[2 * i] << 16) & 0x7fff0000u) * inv,
                                                              __uint_as_float(e8[2 * i] & 0x7fff0000u) * inv, 0, false);
                q[i] = (unsigned)__builtin_amdgcn_cvt_pk_fp8_f32(__uint_as_float((e8[2 * i + 1] << 16) & 0x7fff0000u) * inv,
                                                                 __uint_as_float(e8[2 * i + 1] & 0x7fff0000u) * inv, w, true);
            }
            if (mine) *(LAS u64*)(cp + lane * 8) = ((u64)q[1] << 32) | q[0];
        } else {
            wait_scale();
        }
        while (!fail) {
            asm volatile("" ::: "memory");
            bool ok = true;
#pragma unroll
            for (int h = 0; h < 2; ++h) {
                v[h] = __builtin_amdgcn_raw_buffer_load_b128(rrcv, voff[h], 0, AUX_SYS);
                const unsigned d = ((v[h][0] ^ tagbits) | (v[h][1] ^ tagbits) | (v[h][2] ^ tagbits) | (v[h][3] ^ tagbits)) & 0x80808080u;
                ok &= voff[h] == 0x10000 || d == 0u;
            }
            if (__all(ok)) break;
            fail = give_up(tmo, call, t0, polls);
        }
        STAMP(7);
#pragma unroll
        for (int h = 0; h < 2; ++h) *(LAS u32x4*)(cp + EARLY + h * 1024 + lane * 16) = v[h];
        WAIT_LGKM0();
#pragma unroll
        for (int h = 0; h < 2; ++h) {
            u32x4 o;
#pragma unroll
            for (int j = 0; j < 4; ++j) {
                const unsigned a4 = adr[h][j] & ~3u;
                const unsigned lo = *(LAS unsigned*)(cp + a4), hi = *(LAS unsigned*)(cp + 4 + a4);
                const unsigned win = __builtin_amdgcn_alignbyte(hi, lo, adr[h][j] & 3u);
                o[j] = (__builtin_amdgcn_perm(0u, win, sel[h][j]) & 0x7f7f7f7fu) | (fail ? 0x7f7f7f7fu : 0u);
            }
            __builtin_amdgcn_raw_buffer_store_b128(o, ro, out0 + h * 8 * NL, 0, AUX_SYS);
        }
#pragma unroll
        for (int h = 0; h < 2; ++h)
            __builtin_amdgcn_raw_buffer_store_b128((u32x4){0u, 0u, 0u, 0u}, rrcv, voff[h], 0, AUX_SYS);
        asm volatile(".Lga_end_b:");
    }
    if (threadIdx.x == 0) __hip_atomic_store(cnt, call, RLX_AGENT);
    STAMP(12);
    asm volatile(".Lga_end_c:");
#endif
}

}

extern "C" void kernel_launch(int rank, int nranks, ncclComm_t comm, hipStream_t stream,
                              void* const* d_in, const int* in_sizes, int n_in,
                              void* d_out, int out_size, const int* in_dtypes, int out_dtype,
                              void* const* d_out_all, void* const* d_ws_all, size_t ws_bytes) {
    (void)comm; (void)in_dtypes; (void)out_dtype; (void)d_out_all;
    if (nranks != ga::NR || (unsigned)rank >= (unsigned)ga::NR || n_in < 2 || in_sizes[0] != ga::ML * ga::KK ||
        in_sizes[1] != ga::KK * ga::NN || out_size != ga::NR * ga::ML * ga::NL || ws_bytes < ga::WS_END) {
        fprintf(stderr, "kernel_launch: unexpected problem shape\n");
        return;
    }
    HIP_CHECK(hipFuncSetAttribute((const void*)ga::gemm_a2a_kernel, hipFuncAttributeMaxDynamicSharedMemorySize, ga::LDS_BYTES));
    ga::Params p;
    p.x = d_in[0];
    p.w = d_in[1];
    for (int r = 0; r < ga::NR; ++r) p.ws[r] = (unsigned char*)d_ws_all[r];
    p.myws = p.ws[rank];
    p.myout = (unsigned char*)d_out;
    p.rank = rank;
    p.pad = 0;
    hipLaunchKernelGGL(ga::gemm_a2a_kernel, dim3(ga::GRID), dim3(ga::THREADS), ga::LDS_BYTES, stream, p);
    HIP_CHECK(hipGetLastError());
}
```

```cpp
#include <hip/hip_runtime.h>
#include <rccl/rccl.h>
#include <cstdio>
#include <type_traits>

#define HIP_CHECK(expr) do { hipError_t _e = (expr); if (_e != hipSuccess) { \
    fprintf(stderr, "HIP error %s at %s:%d\n", hipGetErrorString(_e), __FILE__, __LINE__); \
    return; } } while (0)

#define NP2_STEPS 12
#define FETCH_SLOT_ 9
#define GPS_ 5
#define LOADER_PRIO 0
#define LEAD1 3
#define LEAD2 4
#define PRE_MM 4
#define NREP_ 8
#define CODE_PF 3
#define SLOWFIX 2
#define PFA_S 5
#define PFA_T 10
#define PFB_S 13
#define ORD1 1
#define ORD2 1
#define RAMP_ 0
#define LOAD_AUX 0

namespace ga {

#define LAS __attribute__((address_space(3)))
#define GAS __attribute__((address_space(1)))
typedef __bf16 bf16x4 __attribute__((ext_vector_type(4)));
typedef __bf16 bf16x8 __attribute__((ext_vector_type(8)));
typedef float f32x4 __attribute__((ext_vector_type(4)));
typedef unsigned u32x4 __attribute__((ext_vector_type(4)));
typedef unsigned long long u64;

constexpr int NR = 8;
constexpr int ML = 512, NN = 4096, KK = 4096, NL = 512;
constexpr int BK = 64, NKS = KK / BK;
constexpr int NP2 = NP2_STEPS;
constexpr int NP1 = (NKS - NP2) / 2;
constexpr int NS = NP1 + NP2;
static_assert(2 * NP1 + NP2 == NKS && NP2 >= 6, "K split");
constexpr int ST = 4, STAGE_B = 32768, W_OFF = 16384;
constexpr int KB_OFF = ST * STAGE_B;
constexpr int FL_OFF = KB_OFF + 16384;
constexpr int LDS_BYTES = FL_OFF + 512;
constexpr int F_WMAX = 8, F_AMAX = 16, F_CALL = 18, F_BAD = 19;
constexpr int GRID = 256, THREADS = 512, RAMP = RAMP_;
static_assert(RAMP + 4 <= NP1, "prefetch ramp");
constexpr int GPS = GPS_, FETCH_SLOT = FETCH_SLOT_;
static_assert((GPS == 5 || GPS == 10) && FETCH_SLOT >= 2 && FETCH_SLOT < NP2 - 1, "hand-off schedule");

constexpr size_t WS_CNT = 0x0, WS_TMO = 0x1000, WS_LMAX = 0x2000, WS_GMAX = 0x4000  ,
                 WS_STAMP = 0x10000, WS_XCH = 0x100000,
                 WS_MASK = WS_XCH + (size_t)GRID * 32768,
                 WS_PAY = WS_MASK + 2 * (size_t)GRID * 2048,
                 WS_END = WS_PAY + (size_t)GRID * 8192;
constexpr int NREP = NREP_;
constexpr u64 DEADLINE_TICKS = 50000000ull;

constexpr int key1(int q, int t) {
    const int owner = (ORD1 & 1) && t == NP1 - 1 - ((NP1 - 4) & 3) ? 0 : (t & 3);
    if ((ORD1 & 2) && t == q) return -1;
    return owner == q && t > LEAD1 ? 2 * (t - LEAD1) - 1 : 2 * t;
}
constexpr int key2(int q, int t) {
    if (ORD2 == 0) return (t & 7) == q && t > LEAD2 ? 2 * (t - LEAD2) - 1 : 2 * t;
    if (t < 8) return t == q ? -2 : t == ((q + 1) & 7) ? -1 : 2 * t;
    return ((t - 8) & 3) == (q >> 1) ? 2 * (t - LEAD2) - 1 : 2 * t;
}
constexpr u64 order_word(int phase, int q, int w) {
    const int nt = phase == 1 ? NP1 : NP2;
    u64 r = 0;
    for (int t = 0; t < nt; ++t) {
        int pos = 0;
        for (int o = 0; o < nt; ++o) {
            const int ko = phase == 1 ? key1(q, o) : key2(q, o), kt = phase == 1 ? key1(q, t) : key2(q, t);
            if (ko < kt || (ko == kt && o < t)) ++pos;
        }
        if (pos / 12 == w) r |= (u64)t << (5 * (pos % 12));
    }
    return r;
}
static_assert(NP1 <= 32 && NP1 <= 36 && NP2 <= 24, "order tables");

struct Params {
    const void* x;
    const void* w;
    unsigned char* myws;
    unsigned char* myout;
    int rank;
    int pad;
    unsigned char* ws[NR];
};

#define RLX_AGENT __ATOMIC_RELAXED, __HIP_MEMORY_SCOPE_AGENT
#define RLX_SYS __ATOMIC_RELAXED, __HIP_MEMORY_SCOPE_SYSTEM
#define BARRIER() asm volatile("s_barrier" ::: "memory")
#define WAIT_VM(n) asm volatile("s_waitcnt vmcnt(" #n ")" ::: "memory")
#define WAIT_LGKM0() asm volatile("s_waitcnt lgkmcnt(0)" ::: "memory")
#define RFL(x) __builtin_amdgcn_readfirstlane(x)

#define SRC_SLOT(d) rank
#define OUT_PTR p.myout
#define OUT_ROW0 (grp * ML)
#define STAMP(i) do {} while (0)

#if defined(__HIP_DEVICE_COMPILE__)
constexpr int OOB = 0x7fffff00;
constexpr int AUX_SYS = 17;
__device__ __forceinline__ bool give_up(GAS unsigned* tmo, unsigned call, u64 t0, unsigned& polls) {
    if ((++polls & 31u) != 0u) return false;
    if (RFL(__hip_atomic_load(tmo, RLX_AGENT)) == call || __builtin_amdgcn_s_memrealtime() - t0 > DEADLINE_TICKS) {
        __hip_atomic_store(tmo, call, RLX_AGENT);
        return true;
    }
    return false;
}
typedef float f32x2 __attribute__((ext_vector_type(2)));
typedef __bf16 bf16x2 __attribute__((ext_vector_type(2)));
__device__ __forceinline__ unsigned pack2(float a, float b) {
    return __builtin_bit_cast(unsigned, __builtin_convertvector((f32x2){a, b}, bf16x2));
}
__device__ __forceinline__ u32x4 pack_bf16(f32x4 a, f32x4 b) {
    return (u32x4){pack2(a[0], a[1]), pack2(a[2], a[3]), pack2(b[0], b[1]), pack2(b[2], b[3])};
}
__device__ __forceinline__ unsigned umax(unsigned a, unsigned b) { return a > b ? a : b; }
__device__ __forceinline__ unsigned umin(unsigned a, unsigned b) { return a < b ? a : b; }
__device__ __forceinline__ unsigned wave_scan(unsigned v) {
    v += (unsigned)__builtin_amdgcn_update_dpp(0, (int)v, 0x111, 0xF, 0xF, true);
    v += (unsigned)__builtin_amdgcn_update_dpp(0, (int)v, 0x112, 0xF, 0xF, true);
    v += (unsigned)__builtin_amdgcn_update_dpp(0, (int)v, 0x114, 0xF, 0xF, true);
    v += (unsigned)__builtin_amdgcn_update_dpp(0, (int)v, 0x118, 0xF, 0xF, true);
    v += (unsigned)__builtin_amdgcn_update_dpp(0, (int)v, 0x142, 0xA, 0xF, false);
    v += (unsigned)__builtin_amdgcn_update_dpp(0, (int)v, 0x143, 0xC, 0xF, false);
    return v;
}
__device__ __forceinline__ unsigned spread4(unsigned nib) { return (nib * 0x00204081u) & 0x01010101u; }
__device__ __forceinline__ unsigned wave_umax(unsigned v) {
    v = umax(v, (unsigned)__builtin_amdgcn_update_dpp(0, (int)v, 0xB1, 0xF, 0xF, true));
    v = umax(v, (unsigned)__builtin_amdgcn_update_dpp(0, (int)v, 0x4E, 0xF, 0xF, true));
    v = umax(v, (unsigned)__builtin_amdgcn_update_dpp(0, (int)v, 0x141, 0xF, 0xF, true));
    v = umax(v, (unsigned)__builtin_amdgcn_update_dpp(0, (int)v, 0x140, 0xF, 0xF, true));
    return umax(umax(__builtin_amdgcn_readlane(v, 0), __builtin_amdgcn_readlane(v, 16)),
                umax(__builtin_amdgcn_readlane(v, 32), __builtin_amdgcn_readlane(v, 48)));
}
#endif

__global__ __launch_bounds__(THREADS, 2) void gemm_a2a_kernel(Params p) {
#if defined(__HIP_DEVICE_COMPILE__)
    extern __shared__ __attribute__((aligned(16))) char shm_raw[];
    LAS char* shm = (LAS char*)shm_raw;
    typedef u64 u64x2 __attribute__((ext_vector_type(2)));
    u64x2 xw;
    asm volatile("s_load_dwordx4 %0, %1, 0x0" : "=s"(xw) : "s"(__builtin_amdgcn_kernarg_segment_ptr()));
    __builtin_amdgcn_sched_barrier(0);
    const int b = blockIdx.x;
    const int grp = b & 7, idx = b >> 3;
    const int tn = idx & 3, tm = (idx >> 2) & 3, hb = idx >> 4;
    const int wid = RFL(threadIdx.x >> 6);
    const int lane = threadIdx.x & 63;
    auto role_rsrc = [&](int lw) {
        return __builtin_amdgcn_make_buffer_rsrc((void*)((lw < 2 ? xw[1] : xw[0]) - 4096), 0,
                                                 (lw < 2 ? KK * NN * 2 : ML * KK * 2) + 4096, 0x00020000);
    };
    auto role_off = [&](int lw, int i) {
        const int mw = (lw - 2) >> 31;
        const int o = (mw & ((grp * NL + tn * 128 + (lw & 1) * 64) * 2)) | (~mw & ((tm * 128 + (lw & 1) * 64) * (KK * 2)));
        const int sw = lw < 2 ? ((lane >> 3) & 2) : (lane >> 4);
        return (lane >> 3) * (KK * 2) + (((lane & 7) ^ sw ^ ((i & 1) << 2)) << 4) + o + i * (8 * KK * 2) - (i & 3) * 1024 + 4096;
    };
    static_assert(KK == NN, "x and W rows have the same pitch");
    {
        const int lw = wid >> 1, g = wid & 1;
        int so = ((hb * NP1 + ((ORD1 & 2) ? ((tm + tn) & 3) : 0)) * (BK * 2)) << (lw < 2 ? 12 : 0);
        LAS char* dst = shm + (lw < 2 ? W_OFF : 0) + (lw & 1) * 8192 + g * 4096;
        int e = role_off(lw, 0) + g * (32 * KK * 2), o = role_off(lw, 1) + g * (32 * KK * 2);
        asm volatile("s_waitcnt lgkmcnt(0)" : "+s"(xw), "+v"(e), "+v"(o), "+s"(so), "+s"(dst));
        const auto rs0 = role_rsrc(lw);
        __builtin_amdgcn_raw_ptr_buffer_load_lds(rs0, dst, 16, e, so, 0, LOAD_AUX);
        __builtin_amdgcn_raw_ptr_buffer_load_lds(rs0, dst, 16, o, so, 1024, LOAD_AUX);
        __builtin_amdgcn_raw_ptr_buffer_load_lds(rs0, dst, 16, e + 16 * KK * 2 - 2048, so, 2048, LOAD_AUX);
        __builtin_amdgcn_raw_ptr_buffer_load_lds(rs0, dst, 16, o + 16 * KK * 2 - 2048, so, 3072, LOAD_AUX);
        __builtin_amdgcn_sched_barrier(0);
        asm volatile("" ::: "memory");
    }
    auto code_prefetch = [&]() {
        u64 pc;
        int len;
        asm volatile("s_getpc_b64 %0\n.Lga_here%=:\n\ts_mov_b32 %1, .Lga_end_a-.Lga_here%=\n\t"
                     "s_max_i32 %1, %1, .Lga_end_b-.Lga_here%=\n\ts_max_i32 %1, %1, .Lga_end_c-.Lga_here%="
                     : "=s"(pc), "=s"(len) :: "scc");
        auto rc = __builtin_amdgcn_make_buffer_rsrc((void*)pc, 0, len, 0x00020000);
        __builtin_amdgcn_raw_ptr_buffer_load_lds(rc, shm + FL_OFF + 256, 4, lane < 8 ? (idx * 8 + lane) * 128 : OOB, 0, 0, 0);
    };
    if (wid == 0) code_prefetch();
    int rank = p.rank & 7;
    unsigned char* peer_ws = p.ws[grp];
    unsigned char* out_ptr = OUT_PTR;
    asm volatile("" : "+s"(rank), "+s"(peer_ws), "+s"(out_ptr));
    GAS unsigned char* myws = (GAS unsigned char*)p.myws;
    GAS unsigned* tmo = (GAS unsigned*)(myws + WS_TMO);
    GAS unsigned* cnt = (GAS unsigned*)(myws + WS_CNT) + b;
    STAMP(0);

    volatile LAS unsigned* fl = (volatile LAS unsigned*)(shm + FL_OFF);
    auto r3_off = [&](int c, int k) { return c * STAGE_B + W_OFF + k * 1024; };

    f32x4 fin[8];
    const int wr = (wid >> 1) & 1, wc = wid & 1;
    unsigned char* pw = nullptr;

    if (wid >= 4) {
        if (LOADER_PRIO) __builtin_amdgcn_s_setprio(LOADER_PRIO);
        const int lw = wid - 4;
        const bool isw = lw < 2;
        const auto rs = role_rsrc(lw);
        int vo[8];
#pragma unroll
        for (int i = 0; i < 8; ++i) vo[i] = role_off(lw, i);
        const int lds0 = (isw ? W_OFF : 0) + (lw & 1) * 8192, shl = isw ? 12 : 0;
        auto issue4 = [&](int s, int so, int g) {
            LAS char* dst = shm + (s & (ST - 1)) * STAGE_B + lds0 + g * 4096;
            __builtin_amdgcn_raw_ptr_buffer_load_lds(rs, dst, 16, vo[g * 4 + 0], so, 0, LOAD_AUX);
            __builtin_amdgcn_raw_ptr_buffer_load_lds(rs, dst, 16, vo[g * 4 + 1], so, 1024, LOAD_AUX);
            __builtin_amdgcn_raw_ptr_buffer_load_lds(rs, dst, 16, vo[g * 4 + 2], so, 2048, LOAD_AUX);
            __builtin_amdgcn_raw_ptr_buffer_load_lds(rs, dst, 16, vo[g * 4 + 3], so, 3072, LOAD_AUX);
        };
        unsigned c0 = 0;
        if (lw == 3) {
            c0 = __hip_atomic_load(cnt, RLX_AGENT);
            pw = p.ws[lane >> 3];
        }
        int ord;
        {
            const int c1 = (tm + tn) & 3, c2 = (tm * 2 + hb + 2 * tn) & 7;
#define SEL4(f, a) (c1 == 0 ? f(1, 0, a) : c1 == 1 ? f(1, 1, a) : c1 == 2 ? f(1, 2, a) : f(1, 3, a))
#define SEL8(f, a) (c2 < 4 ? (c2 == 0 ? f(2, 0, a) : c2 == 1 ? f(2, 1, a) : c2 == 2 ? f(2, 2, a) : f(2, 3, a)) \
                           : (c2 == 4 ? f(2, 4, a) : c2 == 5 ? f(2, 5, a) : c2 == 6 ? f(2, 6, a) : f(2, 7, a)))
#define OW(ph, q, w) std::integral_constant<u64, order_word(ph, q, w)>::value
            const bool p2 = lane >= NP1;
            const int pos = p2 ? lane - NP1 : lane;
            const u64 a0 = SEL4(OW, 0), a1 = SEL4(OW, 1), a2 = SEL4(OW, 2), b0 = SEL8(OW, 0), b1 = SEL8(OW, 1);
            const u64 wd = p2 ? (pos < 12 ? b0 : b1) : (pos < 12 ? a0 : pos < 24 ? a1 : a2);
            const int t = (int)(wd >> (5 * (pos < 12 ? pos : pos < 24 ? pos - 12 : (pos - 24) & 7))) & 31;
            ord = (((t + (p2 ? 2 * NP1 : hb * NP1)) * (BK * 2)) << shl) + ((p2 && !isw) ? hb * 64 * (KK * 2) : 0);
        }
#define VMW(n) asm volatile("s_waitcnt vmcnt(%0)" :: "i"(n) : "memory")
        const int sl = (3 - (int)(xw[1] >> 7)) & 7, q1 = (tm + tn) & 3;
        const int g12 = tm * 3 + tn - (tn > (sl >> 1) ? 1 : 0), l12 = lane * 12 + g12, l24 = lane * 24 + hb * 12 + g12;
        const int pfo = grp * (NL * 2) + sl * 128 + 4096;
        const bool pfw = SLOWFIX && lw == 0 && tn != (sl >> 1);
        auto prefetch = [&](int row, bool on) {
            __builtin_amdgcn_raw_ptr_buffer_load_lds(rs, shm + FL_OFF + 256, 4, on ? row * (NN * 2) + pfo : OOB, 0, 0, 0);
        };
        auto issue8 = [&](int s) {
            const int so = __builtin_amdgcn_readlane(ord, s);
            if (pfw) {
                if (s == q1 + PFA_S) prefetch((hb * NP1 + PFA_T) * BK + l12, true);
                if (s == q1 + PFB_S) {
                    prefetch((hb * NP1 + PFA_T + 12) * BK + l12, l12 < (NP1 - PFA_T - 12) * BK);
                    if (SLOWFIX > 1) prefetch(2 * NP1 * BK + l24, l24 < NP2 * BK);
                }
            }
            issue4(s, so, 0);
            issue4(s, so, 1);
        };
        issue8(1);
        VMW(8);
        for (int s = 0; s < RAMP; ++s) {
            BARRIER();
            issue8(s + 2);
            VMW(8);
        }
        const int nst = (lw == 3) ? NP1 : NS;
        BARRIER();
        issue8(RAMP + 2);
        issue8(RAMP + 3);
        VMW(16);
        for (int s = RAMP + 1; s < nst - 3; ++s) {
            BARRIER();
            issue8(s + 3);
            VMW(16);
        }
        BARRIER();
        VMW(8);
        BARRIER();
        VMW(0);
        if (lw != 3) {
            BARRIER();
        } else {
            const unsigned call = RFL(c0) + 1;
            fl[F_CALL] = call;
            fl[F_AMAX + 1] = 0;
            fl[F_BAD] = 0;
            asm volatile("" : "+v"(pw));
            code_prefetch();
            WAIT_LGKM0();
            for (int s = NP1 - 1; s < NS; ++s) BARRIER();
        }
    } else {
        const int gq = lane >> 4, qq = (lane & 15) >> 2, pp = lane & 3;
        LAS char* bx[2][2][2];
        LAS char* bw[2][4];
#pragma unroll
        for (int hi = 0; hi < 2; ++hi) {
#pragma unroll
            for (int ks = 0; ks < 2; ++ks) {
                const int xo = hi * 65536 + (lane & 15) * 128 + (((4 * ks + gq) ^ ((lane >> 1) & 7)) << 4);
                bx[hi][ks][0] = shm + xo + wr * 12288;
                bx[hi][ks][1] = shm + xo + 4096 + wr * 4096;
                asm volatile("" : "+v"(bx[hi][ks][0]), "+v"(bx[hi][ks][1]));
            }
#pragma unroll
            for (int j = 0; j < 4; ++j) {
                bw[hi][j] = shm + hi * 65536 + W_OFF + wc * 8192 + 128 * (8 * gq + qq) + ((j ^ ((qq >> 1) | ((gq & 1) << 1))) << 5) + 8 * pp;
                asm volatile("" : "+v"(bw[hi][j]));
            }
        }
        f32x4 acc[4][4];
#pragma unroll
        for (int k = 0; k < 16; ++k) acc[k >> 2][k & 3] = (f32x4){0.f, 0.f, 0.f, 0.f};
        struct Frag { bf16x8 x[4], w[4]; };
        auto rd = [&](Frag& f, int sl, int ks, int ni) {
#pragma unroll
            for (int j = 0; j < 4; ++j) {
                if (j < ni) f.x[j] = *(LAS bf16x8*)(bx[sl >> 1][ks][j >> 1] + (sl & 1) * STAGE_B + (j & 1) * 2048);
                LAS char* wb = bw[sl >> 1][j] + (sl & 1) * STAGE_B + ks * 4096;
                const bf16x4 lo = __builtin_amdgcn_ds_read_tr16_b64_v4bf16((LAS bf16x4*)wb);
                const bf16x4 hi = __builtin_amdgcn_ds_read_tr16_b64_v4bf16((LAS bf16x4*)(wb + 512));
                f.w[j] = __builtin_shufflevector(lo, hi, 0, 1, 2, 3, 4, 5, 6, 7);
            }
        };
        auto mm = [&](const Frag& f, int n0, int n1) {
#pragma unroll
            for (int n = n0; n < n1; ++n)
                acc[n >> 2][n & 3] = __builtin_amdgcn_mfma_f32_16x16x32_bf16(f.w[n & 3], f.x[n >> 2], acc[n >> 2][n & 3], 0, 0, 0);
        };
#define SGB(m, n) __builtin_amdgcn_sched_group_barrier(m, n, 0)
#define SCHED_FENCE() __builtin_amdgcn_sched_barrier(0)
#define GROUP1_END() do { _Pragma("unroll") for (int _i = 0; _i < 12; ++_i) { SGB(0x8, 1); SGB(0x100, 1); } SGB(0x8, 4); SCHED_FENCE(); } while (0)
#define GROUP2_END() do { _Pragma("unroll") for (int _i = 0; _i < 5; ++_i) { SGB(0x8, 1); SGB(0x100, 2); } SGB(0x8, 3); SCHED_FENCE(); } while (0)
        Frag fa, fb;
        auto stage_barrier = [&](int s, int npre) {
            __builtin_amdgcn_s_waitcnt(0xC07F);
            SCHED_FENCE();
            if (npre) mm(fb, 0, npre);
            SCHED_FENCE();
            BARRIER();
            SCHED_FENCE();
        };
        WAIT_VM(0);
        stage_barrier(0, 0);
        STAMP(11);
        rd(fa, 0, 0, 4);
        SCHED_FENCE();
        rd(fb, 0, 1, 4); mm(fa, 0, 16); GROUP1_END();
        auto stage1 = [&](int s, int sl) {
            stage_barrier(s, PRE_MM);
            rd(fa, sl, 0, 4); mm(fb, PRE_MM, 16); GROUP1_END();
            rd(fb, sl, 1, 4); mm(fa, 0, 16); GROUP1_END();
        };
        for (int s = 1; s + 3 < NP1; s += 4) { stage1(s, 1); stage1(s + 1, 2); stage1(s + 2, 3); stage1(s + 3, 0); }
#pragma unroll
        for (int s = NP1 - (NP1 - 1) % 4; s < NP1; ++s) stage1(s, s & 3);
#pragma unroll
        for (int k = 0; k < 4; ++k) bx[k >> 1][k & 1][0] -= wr * 8192;
        stage_barrier(NP1, PRE_MM);
        rd(fa, NP1 & 3, 0, 2);
        mm(fb, PRE_MM, 16);
        SCHED_FENCE();
        STAMP(1);
        const bool giver = (wr != hb);
        const unsigned call = RFL(fl[F_CALL]), xtag = (call % 65535u + 1u) << 16;
        u32x4 pk[10];
        if (giver) {
#pragma unroll
            for (int g = 0; g < 10; ++g) {
                auto val = [&](int i) { const int v = (g % 5) * 7 + i; return v < 32 ? acc[(g / 5) * 2 + (v >> 4)][(v >> 2) & 3][v & 3] : 0.f; };
                pk[g] = (u32x4){pack2(val(0), val(1)), pack2(val(2), val(3)), pack2(val(4), val(5)), (pack2(val(6), 0.f) & 0xffffu) | xtag};
            }
#pragma unroll
            for (int k = 0; k < 8; ++k) acc[k >> 2][k & 3] = (f32x4){0.f, 0.f, 0.f, 0.f};
        }
        auto rdump = __builtin_amdgcn_make_buffer_rsrc((void*)(p.myws + WS_XCH + (size_t)b * 32768), 0, 32768, 0x00020000);
        auto rp = __builtin_amdgcn_make_buffer_rsrc((void*)(p.myws + WS_XCH + (size_t)(b ^ 128) * 32768), 0, 32768, 0x00020000);
        SCHED_FENCE();
        u32x4 r2[5];
        auto fetch = [&]() {
#pragma unroll
            for (int g = 0; g < 5; ++g) r2[g] = __builtin_amdgcn_raw_buffer_load_b128(rp, lane * 16, ((wc * 2 + wr) * 5 + g) * 1024, AUX_SYS);
        };
        auto fresh = [&]() { return __all((((r2[0][3] ^ xtag) | (r2[1][3] ^ xtag) | (r2[2][3] ^ xtag) | (r2[3][3] ^ xtag) | (r2[4][3] ^ xtag)) >> 16) == 0u); };
        auto add_r2 = [&]() {
#pragma unroll
            for (int v = 0; v < 32; ++v) {
                const unsigned d = r2[v / 7][(v % 7) >> 1];
                acc[v >> 4][(v >> 2) & 3][v & 3] += __uint_as_float(((v % 7) & 1) ? (d & 0xffff0000u) : (d << 16));
            }
        };
        bool r2_in = false;
#pragma unroll
        for (int u = 0; u < NP2; ++u) {
            if (u > 0) { stage_barrier(NP1 + u, 0); rd(fa, (NP1 + u) & 3, 0, 2); mm(fb, 0, 8); GROUP2_END(); }
            if (u == NP2 - 1) {
                r2_in = fresh();
                if (r2_in) add_r2();
                SCHED_FENCE();
            }
            rd(fb, (NP1 + u) & 3, 1, 2); mm(fa, 0, 8); GROUP2_END();
            if (u * GPS < 10) {
                if (giver) {
#pragma unroll
                    for (int g = u * GPS; g < (u + 1) * GPS; ++g)
                        __builtin_amdgcn_raw_buffer_store_b128(pk[g], rdump, lane * 16, ((wc * 2 + ((g / 5) ^ wr)) * 5 + g % 5) * 1024, AUX_SYS);
                    asm volatile("s_nop 1" ::: "memory");
                } else {
#pragma unroll
                    for (int k = u * (GPS * 8 / 10); k < (u + 1) * (GPS * 8 / 10); ++k)
                        *(LAS f32x4*)(shm + KB_OFF + wc * 8192 + k * 1024 + lane * 16) = acc[2 + (k >> 2)][k & 3];
                }
                SCHED_FENCE();
            }
            if (u == 3 && giver) {
#pragma unroll
                for (int k = 0; k < 8; ++k) acc[k >> 2][k & 3] += *(LAS f32x4*)(shm + KB_OFF + wc * 8192 + k * 1024 + lane * 16);
                SCHED_FENCE();
            }
            if (u == FETCH_SLOT) { fetch(); SCHED_FENCE(); }
        }
        mm(fb, 0, 8);
        if (!r2_in) {
            STAMP(8);
            const u64 t0 = __builtin_amdgcn_s_memrealtime();
            unsigned polls = 0;
            for (;;) {
                asm volatile("" ::: "memory");
                fetch();
                if (fresh()) break;
                if (give_up(tmo, call, t0, polls)) { fl[F_BAD] = 1; break; }
            }
            add_r2();
        }
        STAMP(2);
        float mx = 0.f;
#pragma unroll
        for (int k = 0; k < 8; ++k) {
            fin[k] = acc[k >> 2][k & 3];
            mx = fmaxf(fmaxf(mx, fin[k][0]), fin[k][1]);
            mx = fmaxf(fmaxf(mx, fin[k][2]), fin[k][3]);
        }
        fl[F_WMAX + wid] = wave_umax(__float_as_uint(mx) & 0x7fffffffu);
        WAIT_LGKM0();
    }
    BARRIER();
    STAMP(3);
    const unsigned call = RFL(fl[F_CALL]);
    if (wid < 4) {
#pragma unroll
        for (int k = 0; k < 8; ++k) {
#pragma unroll
            for (int e = 0; e < 4; ++e) fin[k][e] = fmaxf(fin[k][e], 0.f);
            *(LAS f32x4*)(shm + r3_off(wc, wr * 8 + k) + ((lane ^ ((k & 3) << 2)) << 4)) = fin[k];
        }
        WAIT_LGKM0();
    } else if (wid == 7) {
        const u32x4 m4 = *(volatile LAS u32x4*)(shm + FL_OFF + F_WMAX * 4);
        const u64 gran = ((u64)call << 32) | umax(umax(m4[0], m4[1]), umax(m4[2], m4[3]));
        if (lane == 0) __hip_atomic_store((GAS u64*)(myws + WS_LMAX) + b, gran, RLX_AGENT);
    }
    BARRIER();
    const bool remote = grp != rank;
    const unsigned tagbits = spread4(call % 15u + 1u) << 7;
    const unsigned tag16 = call % 65535u + 1u;
    LAS char* cp = shm + 2 * STAGE_B + wid * 8192;
    auto ro = __builtin_amdgcn_make_buffer_rsrc((void*)out_ptr, 0, NR * ML * NL, 0x00020000);
    const int out0 = (OUT_ROW0 + tm * 128 + hb * 64 + (wid & 3) * 16 + (lane >> 3)) * NL + tn * 128 + (lane & 7) * 16;
    auto publish_scale = [&](unsigned am, bool bad) {
        const float amax = __uint_as_float(am);
        float inv = __builtin_amdgcn_rcpf(amax * (1.0f / 448.0f));
        if (!(amax > 1e-30f && amax < 1e30f)) inv = amax > 0.f ? 1.0f / (amax / 448.0f) : 0.f;
        *(volatile LAS u64*)(shm + FL_OFF + F_AMAX * 4) = ((u64)(bad ? 2u : 1u) << 32) | __float_as_uint(inv);
    };
    if (wid == 7) {
        const u64 t0 = __builtin_amdgcn_s_memrealtime();
        unsigned polls = 0;
        const size_t goff = WS_GMAX + (size_t)(call & 1) * (NREP * 128);
        if (b == rank) {
            GAS u64* lmax = (GAS u64*)(myws + WS_LMAX);
            unsigned gm;
            bool bad = false;
            for (;;) {
                bool ok = true;
                gm = 0;
#pragma unroll
                for (int k = 0; k < 4; ++k) {
                    const u64 v = __hip_atomic_load(lmax + k * 64 + lane, RLX_AGENT);
                    ok &= (unsigned)(v >> 32) >= call;
                    gm = umax(gm, (unsigned)v);
                }
                if (__all(ok)) break;
                if (give_up(tmo, call, t0, polls)) { bad = true; break; }
            }
            gm = wave_umax(gm);
            if (!bad) {
#pragma unroll
                for (int k = 0; k < NREP / 8; ++k)
                    __hip_atomic_store((GAS u64*)(pw + goff + (k * 8 + (lane & 7)) * 128) + SRC_SLOT(lane >> 3), ((u64)call << 32) | gm, RLX_SYS);
            }
            STAMP(4);
        }
        GAS u64* gp = (GAS u64*)(myws + goff + (b & (NREP - 1)) * 128) + (lane & 7);
        unsigned am;
        bool bad = fl[F_BAD] != 0u;
        for (;;) {
            const u64 v = __hip_atomic_load(gp, RLX_SYS);
            am = (unsigned)v;
            if (__all((unsigned)(v >> 32) >= call)) break;
            if (give_up(tmo, call, t0, polls)) { bad = true; break; }
        }
        publish_scale(wave_umax(am), bad);
        STAMP(5);
    }
    auto quant = [&](const f32x4* v, float inv) {
        u32x4 o;
#pragma unroll
        for (int i = 0; i < 4; ++i) {
            const f32x4 t = v[i] * inv;
            const int w = __builtin_amdgcn_cvt_pk_fp8_f32(t[0], t[1], 0, false);
            o[i] = (unsigned)__builtin_amdgcn_cvt_pk_fp8_f32(t[2], t[3], w, true);
        }
        return o;
    };
    auto wait_scale = [&]() {
        u64 v;
        do v = *(volatile LAS u64*)(shm + FL_OFF + F_AMAX * 4); while (RFL((unsigned)(v >> 32)) == 0u);
        return v;
    };
    if (wid < 4) {
        f32x4 y[2][4];
#pragma unroll
        for (int h = 0; h < 2; ++h)
#pragma unroll
            for (int i = 0; i < 4; ++i)
                y[h][i] = *(LAS f32x4*)(shm + r3_off((lane >> 2) & 1, wid * 4 + (lane & 3)) +
                                        (((h * 8 + (lane >> 3) + 16 * i) ^ ((lane & 3) << 2)) << 4));
        int v0 = out0, v1 = out0 + 8 * NL;
        unsigned andb = 0xffffffffu, orb = 0u;
        auto rsnd = ro;
        if (remote) {
            unsigned mask[2] = {0u, 0u};
#pragma unroll
            for (int e = 0; e < 32; ++e) mask[e >> 4] |= (y[e >> 4][(e >> 2) & 3][e & 3] > 0.f ? 1u : 0u) << (e & 15);
            const int slot = idx * 8 + SRC_SLOT(grp);
            GAS unsigned* mo = (GAS unsigned*)((GAS unsigned char*)peer_ws + WS_MASK + ((size_t)(call & 1) * GRID + slot) * 2048) + threadIdx.x;
            __hip_atomic_store(mo, mask[0] | (tag16 << 16), RLX_SYS);
            __hip_atomic_store(mo + 256, mask[1] | (tag16 << 16), RLX_SYS);
            const unsigned c = __builtin_popcount(mask[0]) | (__builtin_popcount(mask[1]) << 16), incl = wave_scan(c);
            const unsigned tot = __builtin_amdgcn_readlane(incl, 63), n = (tot & 0xffffu) + (tot >> 16);
#pragma unroll
            for (int h = 0; h < 2; ++h) {
                unsigned pos = (h ? (tot & 0xffffu) + ((incl - c) >> 16) : ((incl - c) & 0xffffu)) * 4;
#pragma unroll
                for (int e = 0; e < 16; ++e)
                    if (y[h][e >> 2][e & 3] > 0.f) { *(LAS float*)(cp + pos) = y[h][e >> 2][e & 3]; pos += 4; }
            }
            WAIT_LGKM0();
#pragma unroll
            for (int h = 0; h < 2; ++h)
#pragma unroll
                for (int i = 0; i < 4; ++i) y[h][i] = *(LAS f32x4*)(cp + h * 4096 + lane * 64 + i * 16);
            rsnd = __builtin_amdgcn_make_buffer_rsrc((void*)(peer_ws + WS_PAY + (size_t)slot * 8192), 0, 8192, 0x00020000);
            v0 = (unsigned)lane * 16 < n ? wid * 2048 + lane * 16 : OOB;
            v1 = (unsigned)lane * 16 + 1024 < n ? wid * 2048 + 1024 + lane * 16 : OOB;
            andb = 0x7f7f7f7fu;
            orb = tagbits;
        }
        asm volatile("" : "+v"(v0), "+v"(v1), "+s"(rsnd), "+s"(andb), "+s"(orb), "+v"(y[0][0]), "+v"(y[0][1]), "+v"(y[0][2]), "+v"(y[0][3]),
                          "+v"(y[1][0]), "+v"(y[1][1]), "+v"(y[1][2]), "+v"(y[1][3]));
        const u64 sv = wait_scale();
        const float inv = __uint_as_float((unsigned)sv);
        if ((unsigned)(sv >> 32) != 1u) orb |= 0x7f7f7f7fu;
        u32x4 o = quant(y[0], inv);
        __builtin_amdgcn_raw_buffer_store_b128((o & andb) | orb, rsnd, v0, 0, AUX_SYS);
        o = quant(y[1], inv);
        __builtin_amdgcn_raw_buffer_store_b128((o & andb) | orb, rsnd, v1, 0, AUX_SYS);
        STAMP(6);
        asm volatile(".Lga_end_a:");
    } else if (remote) {
        const u64 t0 = __builtin_amdgcn_s_memrealtime();
        unsigned polls = 0;
        bool fail = false;
        GAS unsigned* mi = (GAS unsigned*)(myws + WS_MASK + ((size_t)(call & 1) * GRID + b) * 2048) + (threadIdx.x - 256);
        unsigned mw[2];
        for (;;) {
            mw[0] = __hip_atomic_load(mi, RLX_SYS);
            mw[1] = __hip_atomic_load(mi + 256, RLX_SYS);
            if (__all((mw[0] >> 16) == tag16 && (mw[1] >> 16) == tag16)) break;
            if (give_up(tmo, call, t0, polls)) { fail = true; break; }
        }
        const unsigned c = __builtin_popcount(mw[0] & 0xffffu) | (__builtin_popcount(mw[1] & 0xffffu) << 16), incl = wave_scan(c);
        const unsigned tot = __builtin_amdgcn_readlane(incl, 63), n = (tot & 0xffffu) + (tot >> 16);
        unsigned adr[2][4], sel[2][4];
        int voff[2];
#pragma unroll
        for (int h = 0; h < 2; ++h) {
            const unsigned mask = mw[h] & 0xffffu, p0 = h ? (tot & 0xffffu) + ((incl - c) >> 16) : ((incl - c) & 0xffffu);
            voff[h] = (unsigned)lane * 16 + h * 1024 < n ? (wid - 4) * 2048 + h * 1024 + lane * 16 : OOB;
#pragma unroll
            for (int j = 0; j < 4; ++j) {
                adr[h][j] = p0 + __builtin_popcount(mask & ((1u << (4 * j)) - 1u));
                const unsigned x = spread4((mask >> (4 * j)) & 15u);
                sel[h][j] = ((x * 0x01010100u) & (x * 0xffu)) | ((x ^ 0x01010101u) * 0x0cu);
                asm volatile("" : "+v"(adr[h][j]), "+v"(sel[h][j]));
            }
        }
        auto rrcv = __builtin_amdgcn_make_buffer_rsrc((void*)(p.myws + WS_PAY + (size_t)b * 8192), 0, 8192, 0x00020000);
        u32x4 v[2];
        v[0] = v[1] = (u32x4){0u, 0u, 0u, 0u};
        wait_scale();
        while (!fail) {
            asm volatile("" ::: "memory");
            bool ok = true;
#pragma unroll
            for (int h = 0; h < 2; ++h) {
                v[h] = __builtin_amdgcn_raw_buffer_load_b128(rrcv, voff[h], 0, AUX_SYS);
                const unsigned d = ((v[h][0] ^ tagbits) | (v[h][1] ^ tagbits) | (v[h][2] ^ tagbits) | (v[h][3] ^ tagbits)) & 0x80808080u;
                ok &= voff[h] == OOB || d == 0u;
            }
            if (__all(ok)) break;
            fail = give_up(tmo, call, t0, polls);
        }
        STAMP(7);
#pragma unroll
        for (int h = 0; h < 2; ++h) *(LAS u32x4*)(cp + h * 1024 + lane * 16) = v[h];
        WAIT_LGKM0();
#pragma unroll
        for (int h = 0; h < 2; ++h) {
            u32x4 o;
#pragma unroll
            for (int j = 0; j < 4; ++j) {
                const unsigned a4 = adr[h][j] & ~3u;
                const unsigned lo = *(LAS unsigned*)(cp + a4), hi = *(LAS unsigned*)(cp + 4 + a4);
                const unsigned win = __builtin_amdgcn_alignbyte(hi, lo, adr[h][j] & 3u);
                o[j] = (__builtin_amdgcn_perm(0u, win, sel[h][j]) & 0x7f7f7f7fu) | (fail ? 0x7f7f7f7fu : 0u);
            }
            __builtin_amdgcn_raw_buffer_store_b128(o, ro, out0 + h * 8 * NL, 0, AUX_SYS);
        }
#pragma unroll
        for (int h = 0; h < 2; ++h)
            __builtin_amdgcn_raw_buffer_store_b128((u32x4){0u, 0u, 0u, 0u}, rrcv, voff[h], 0, AUX_SYS);
        asm volatile(".Lga_end_b:");
    }
    if (threadIdx.x == 0) __hip_atomic_store(cnt, call, RLX_AGENT);
    STAMP(12);
    asm volatile(".Lga_end_c:");
#endif
}

}

extern "C" void kernel_launch(int rank, int nranks, ncclComm_t comm, hipStream_t stream,
                              void* const* d_in, const int* in_sizes, int n_in,
                              void* d_out, int out_size, const int* in_dtypes, int out_dtype,
                              void* const* d_out_all, void* const* d_ws_all, size_t ws_bytes) {
    (void)comm; (void)in_dtypes; (void)out_dtype; (void)d_out_all;
    if (nranks != ga::NR || (unsigned)rank >= (unsigned)ga::NR || n_in < 2 || in_sizes[0] != ga::ML * ga::KK ||
        in_sizes[1] != ga::KK * ga::NN || out_size != ga::NR * ga::ML * ga::NL || ws_bytes < ga::WS_END) {
        fprintf(stderr, "kernel_launch: unexpected problem shape\n");
        return;
    }
    HIP_CHECK(hipFuncSetAttribute((const void*)ga::gemm_a2a_kernel, hipFuncAttributeMaxDynamicSharedMemorySize, ga::LDS_BYTES));
    ga::Params p;
    p.x = d_in[0];
    p.w = d_in[1];
    for (int r = 0; r < ga::NR; ++r) p.ws[r] = (unsigned char*)d_ws_all[r];
    p.myws = p.ws[rank];
    p.myout = (unsigned char*)d_out;
    p.rank = rank;
    p.pad = 0;
    hipLaunchKernelGGL(ga::gemm_a2a_kernel, dim3(ga::GRID), dim3(ga::THREADS), ga::LDS_BYTES, stream, p);
    HIP_CHECK(hipGetLastError());
}
```

```cpp
#include <hip/hip_runtime.h>
#include <rccl/rccl.h>
#include <cstdio>
#include <type_traits>

#define HIP_CHECK(expr) do { hipError_t _e = (expr); if (_e != hipSuccess) { \
    fprintf(stderr, "HIP error %s at %s:%d\n", hipGetErrorString(_e), __FILE__, __LINE__); \
    return; } } while (0)

namespace ga {

#define LAS __attribute__((address_space(3)))
#define GAS __attribute__((address_space(1)))
typedef __bf16 bf16x4 __attribute__((ext_vector_type(4)));
typedef __bf16 bf16x8 __attribute__((ext_vector_type(8)));
typedef float f32x4 __attribute__((ext_vector_type(4)));
typedef unsigned u32x4 __attribute__((ext_vector_type(4)));
typedef unsigned long long u64;

constexpr int NR = 8;
constexpr int ML = 512, NN = 4096, KK = 4096, NL = 512;
constexpr int BK = 64, NKS = KK / BK;
constexpr int NP2 = 12;
constexpr int NP1 = (NKS - NP2) / 2;
constexpr int NS = NP1 + NP2;
static_assert(2 * NP1 + NP2 == NKS && NP2 >= 6, "K split");
constexpr int ST = 4, STAGE_B = 32768, W_OFF = 16384;
constexpr int KB_OFF = ST * STAGE_B;
constexpr int FL_OFF = KB_OFF + 16384;
constexpr int LDS_BYTES = FL_OFF + 512;
constexpr int F_WMAX = 8, F_AMAX = 16, F_CALL = 18, F_BAD = 19;
constexpr int GRID = 256, THREADS = 512;
constexpr int PRE_MM = 4;
constexpr int GPS = 5, FETCH_SLOT = 9;
static_assert((GPS == 5 || GPS == 10) && FETCH_SLOT >= 2 && FETCH_SLOT < NP2 - 1, "hand-off schedule");

constexpr size_t WS_CNT = 0x0,
                 WS_TMO = 0x1000,
                 WS_LMAX = 0x2000,
                 WS_GMAX = 0x4000,
                 WS_XCH = 0x100000,
                 WS_MASK = WS_XCH + (size_t)GRID * 32768,
                 WS_PAY = WS_MASK + 2 * (size_t)GRID * 2048,
                 WS_END = WS_PAY + (size_t)GRID * 8192;
constexpr int NREP = 8;
constexpr u64 DEADLINE_TICKS = 150000000ull;

constexpr int LEAD1 = 3, LEAD2 = 4;
constexpr int key1(int q, int t) {
    const int owner = t == NP1 - 1 - ((NP1 - 4) & 3) ? 0 : (t & 3);
    return owner == q && t > LEAD1 ? 2 * (t - LEAD1) - 1 : 2 * t;
}
constexpr int key2(int q, int t) {
    if (t < 8) return t == q ? -2 : t == ((q + 1) & 7) ? -1 : 2 * t;
    return ((t - 8) & 3) == (q >> 1) ? 2 * (t - LEAD2) - 1 : 2 * t;
}
constexpr u64 order_word(int phase, int q, int w) {
    const int nt = phase == 1 ? NP1 : NP2;
    u64 r = 0;
    for (int t = 0; t < nt; ++t) {
        int pos = 0;
        for (int o = 0; o < nt; ++o) {
            const int ko = phase == 1 ? key1(q, o) : key2(q, o), kt = phase == 1 ? key1(q, t) : key2(q, t);
            if (ko < kt || (ko == kt && o < t)) ++pos;
        }
        if (pos / 12 == w) r |= (u64)t << (5 * (pos % 12));
    }
    return r;
}
static_assert(NP1 <= 32 && NP2 <= 24 && (order_word(1, 0, 0) & 31) == 0 && (order_word(1, 3, 0) & 31) == 0, "order tables");

struct Params {
    const void* x;
    const void* w;
    unsigned char* myws;
    unsigned char* myout;
    int rank;
    int pad;
    unsigned char* ws[NR];
};

#define RLX_AGENT __ATOMIC_RELAXED, __HIP_MEMORY_SCOPE_AGENT
#define RLX_SYS __ATOMIC_RELAXED, __HIP_MEMORY_SCOPE_SYSTEM
#define BARRIER() asm volatile("s_barrier" ::: "memory")
#define WAIT_VM(n) asm volatile("s_waitcnt vmcnt(" #n ")" ::: "memory")
#define WAIT_LGKM0() asm volatile("s_waitcnt lgkmcnt(0)" ::: "memory")
#define RFL(x) __builtin_amdgcn_readfirstlane(x)

#if defined(__HIP_DEVICE_COMPILE__)
constexpr int OOB = 0x7fffff00;
constexpr int AUX_SYS = 17;
__device__ __forceinline__ bool give_up(GAS unsigned* tmo, unsigned call, u64 t0, unsigned& polls) {
    if ((++polls & 31u) != 0u) return false;
    if (RFL(__hip_atomic_load(tmo, RLX_AGENT)) == call || __builtin_amdgcn_s_memrealtime() - t0 > DEADLINE_TICKS) {
        __hip_atomic_store(tmo, call, RLX_AGENT);
        return true;
    }
    return false;
}
typedef float f32x2 __attribute__((ext_vector_type(2)));
typedef __bf16 bf16x2 __attribute__((ext_vector_type(2)));
__device__ __forceinline__ unsigned pack2(float a, float b) {
    return __builtin_bit_cast(unsigned, __builtin_convertvector((f32x2){a, b}, bf16x2));
}
__device__ __forceinline__ unsigned umax(unsigned a, unsigned b) { return a > b ? a : b; }
__device__ __forceinline__ unsigned wave_scan(unsigned v) {
    v += (unsigned)__builtin_amdgcn_update_dpp(0, (int)v, 0x111, 0xF, 0xF, true);
    v += (unsigned)__builtin_amdgcn_update_dpp(0, (int)v, 0x112, 0xF, 0xF, true);
    v += (unsigned)__builtin_amdgcn_update_dpp(0, (int)v, 0x114, 0xF, 0xF, true);
    v += (unsigned)__builtin_amdgcn_update_dpp(0, (int)v, 0x118, 0xF, 0xF, true);
    v += (unsigned)__builtin_amdgcn_update_dpp(0, (int)v, 0x142, 0xA, 0xF, false);
    v += (unsigned)__builtin_amdgcn_update_dpp(0, (int)v, 0x143, 0xC, 0xF, false);
    return v;
}
__device__ __forceinline__ unsigned spread4(unsigned nib) { return (nib * 0x00204081u) & 0x01010101u; }
__device__ __forceinline__ unsigned wave_umax(unsigned v) {
    v = umax(v, (unsigned)__builtin_amdgcn_update_dpp(0, (int)v, 0xB1, 0xF, 0xF, true));
    v = umax(v, (unsigned)__builtin_amdgcn_update_dpp(0, (int)v, 0x4E, 0xF, 0xF, true));
    v = umax(v, (unsigned)__builtin_amdgcn_update_dpp(0, (int)v, 0x141, 0xF, 0xF, true));
    v = umax(v, (unsigned)__builtin_amdgcn_update_dpp(0, (int)v, 0x140, 0xF, 0xF, true));
    return umax(umax(__builtin_amdgcn_readlane(v, 0), __builtin_amdgcn_readlane(v, 16)),
                umax(__builtin_amdgcn_readlane(v, 32), __builtin_amdgcn_readlane(v, 48)));
}
#endif

__global__ __launch_bounds__(THREADS, 2) void gemm_a2a_kernel(Params p) {
#if defined(__HIP_DEVICE_COMPILE__)
    extern __shared__ __attribute__((aligned(16))) char shm_raw[];
    LAS char* shm = (LAS char*)shm_raw;
    typedef u64 u64x2 __attribute__((ext_vector_type(2)));
    u64x2 xw;
    asm volatile("s_load_dwordx4 %0, %1, 0x0" : "=s"(xw) : "s"(__builtin_amdgcn_kernarg_segment_ptr()));
    __builtin_amdgcn_sched_barrier(0);
    const int b = blockIdx.x;
    const int grp = b & 7, idx = b >> 3;
    const int tn = idx & 3, tm = (idx >> 2) & 3, hb = idx >> 4;
    const int wid = RFL(threadIdx.x >> 6);
    const int lane = threadIdx.x & 63;
    auto role_rsrc = [&](int lw) {
        return __builtin_amdgcn_make_buffer_rsrc((void*)((lw < 2 ? xw[1] : xw[0]) - 4096), 0,
                                                 (lw < 2 ? KK * NN * 2 : ML * KK * 2) + 4096, 0x00020000);
    };
    auto role_off = [&](int lw, int i) {
        const int mw = (lw - 2) >> 31;
        const int o = (mw & ((grp * NL + tn * 128 + (lw & 1) * 64) * 2)) | (~mw & ((tm * 128 + (lw & 1) * 64) * (KK * 2)));
        const int sw = lw < 2 ? ((lane >> 3) & 2) : (lane >> 4);
        return (lane >> 3) * (KK * 2) + (((lane & 7) ^ sw ^ ((i & 1) << 2)) << 4) + o + i * (8 * KK * 2) - (i & 3) * 1024 + 4096;
    };
    static_assert(KK == NN, "x and W rows have the same pitch");
    {
        const int lw = wid >> 1, g = wid & 1;
        int so = (hb * NP1 * (BK * 2)) << (lw < 2 ? 12 : 0);
        LAS char* dst = shm + (lw < 2 ? W_OFF : 0) + (lw & 1) * 8192 + g * 4096;
        int e = role_off(lw, 0) + g * (32 * KK * 2), o = role_off(lw, 1) + g * (32 * KK * 2);
        asm volatile("s_waitcnt lgkmcnt(0)" : "+s"(xw), "+v"(e), "+v"(o), "+s"(so), "+s"(dst));
        const auto rs0 = role_rsrc(lw);
        __builtin_amdgcn_raw_ptr_buffer_load_lds(rs0, dst, 16, e, so, 0, 0);
        __builtin_amdgcn_raw_ptr_buffer_load_lds(rs0, dst, 16, o, so, 1024, 0);
        __builtin_amdgcn_raw_ptr_buffer_load_lds(rs0, dst, 16, e + 16 * KK * 2 - 2048, so, 2048, 0);
        __builtin_amdgcn_raw_ptr_buffer_load_lds(rs0, dst, 16, o + 16 * KK * 2 - 2048, so, 3072, 0);
        __builtin_amdgcn_sched_barrier(0);
        asm volatile("" ::: "memory");
    }
    auto code_prefetch = [&]() {
        u64 pc;
        int len;
        asm volatile("s_getpc_b64 %0\n.Lga_here%=:\n\ts_mov_b32 %1, .Lga_end_a-.Lga_here%=\n\t"
                     "s_max_i32 %1, %1, .Lga_end_b-.Lga_here%=\n\ts_max_i32 %1, %1, .Lga_end_c-.Lga_here%=\n\ts_max_i32 %1, %1, 0"
                     : "=s"(pc), "=s"(len) :: "scc");
        auto rc = __builtin_amdgcn_make_buffer_rsrc((void*)pc, 0, len, 0x00020000);
        __builtin_amdgcn_raw_ptr_buffer_load_lds(rc, shm + FL_OFF + 256, 4, lane < 8 ? (idx * 8 + lane) * 128 : OOB, 0, 0, 0);
    };
    if (wid == 0) code_prefetch();
    int rank = p.rank & 7;
    unsigned char* peer_ws = p.ws[grp];
    unsigned char* out_ptr = p.myout;
    asm volatile("" : "+s"(rank), "+s"(peer_ws), "+s"(out_ptr));
    GAS unsigned char* myws = (GAS unsigned char*)p.myws;
    GAS unsigned* tmo = (GAS unsigned*)(myws + WS_TMO);
    GAS unsigned* cnt = (GAS unsigned*)(myws + WS_CNT) + b;

    volatile LAS unsigned* fl = (volatile LAS unsigned*)(shm + FL_OFF);
    auto r3_off = [&](int c, int k) { return c * STAGE_B + W_OFF + k * 1024; };

    f32x4 fin[8];
    const int wr = (wid >> 1) & 1, wc = wid & 1;
    unsigned char* pw = nullptr;

    if (wid >= 4) {
        const int lw = wid - 4;
        const bool isw = lw < 2;
        const auto rs = role_rsrc(lw);
        int vo[8];
#pragma unroll
        for (int i = 0; i < 8; ++i) vo[i] = role_off(lw, i);
        const int lds0 = (isw ? W_OFF : 0) + (lw & 1) * 8192, shl = isw ? 12 : 0;
        auto issue4 = [&](int s, int so, int g) {
            LAS char* dst = shm + (s & (ST - 1)) * STAGE_B + lds0 + g * 4096;
            __builtin_amdgcn_raw_ptr_buffer_load_lds(rs, dst, 16, vo[g * 4 + 0], so, 0, 0);
            __builtin_amdgcn_raw_ptr_buffer_load_lds(rs, dst, 16, vo[g * 4 + 1], so, 1024, 0);
            __builtin_amdgcn_raw_ptr_buffer_load_lds(rs, dst, 16, vo[g * 4 + 2], so, 2048, 0);
            __builtin_amdgcn_raw_ptr_buffer_load_lds(rs, dst, 16, vo[g * 4 + 3], so, 3072, 0);
        };
        unsigned c0 = 0;
        if (lw == 3) {
            c0 = __hip_atomic_load(cnt, RLX_AGENT);
            pw = p.ws[lane >> 3];
        }
        int ord;
        {
            const int c1 = (tm + tn) & 3, c2 = (tm * 2 + hb + 2 * tn) & 7;
#define SEL4(f, a) (c1 == 0 ? f(1, 0, a) : c1 == 1 ? f(1, 1, a) : c1 == 2 ? f(1, 2, a) : f(1, 3, a))
#define SEL8(f, a) (c2 < 4 ? (c2 == 0 ? f(2, 0, a) : c2 == 1 ? f(2, 1, a) : c2 == 2 ? f(2, 2, a) : f(2, 3, a)) \
                           : (c2 == 4 ? f(2, 4, a) : c2 == 5 ? f(2, 5, a) : c2 == 6 ? f(2, 6, a) : f(2, 7, a)))
#define OW(ph, q, w) std::integral_constant<u64, order_word(ph, q, w)>::value
            const bool p2 = lane >= NP1;
            const int pos = p2 ? lane - NP1 : lane;
            const u64 a0 = SEL4(OW, 0), a1 = SEL4(OW, 1), a2 = SEL4(OW, 2), b0 = SEL8(OW, 0), b1 = SEL8(OW, 1);
            const u64 wd = p2 ? (pos < 12 ? b0 : b1) : (pos < 12 ? a0 : pos < 24 ? a1 : a2);
            const int t = (int)(wd >> (5 * (pos < 12 ? pos : pos < 24 ? pos - 12 : (pos - 24) & 7))) & 31;
            ord = (((t + (p2 ? 2 * NP1 : hb * NP1)) * (BK * 2)) << shl) + ((p2 && !isw) ? hb * 64 * (KK * 2) : 0);
        }
#define VMW(n) asm volatile("s_waitcnt vmcnt(%0)" :: "i"(n) : "memory")
        const int sl = (3 - (int)(xw[1] >> 7)) & 7, q1 = (tm + tn) & 3;
        const int g12 = tm * 3 + tn - (tn > (sl >> 1) ? 1 : 0), l12 = lane * 12 + g12, l24 = lane * 24 + hb * 12 + g12;
        const int pfo = grp * (NL * 2) + sl * 128 + 4096;
        const bool pfw = lw == 0 && tn != (sl >> 1);
        auto prefetch = [&](int row, bool on) {
            __builtin_amdgcn_raw_ptr_buffer_load_lds(rs, shm + FL_OFF + 256, 4, on ? row * (NN * 2) + pfo : OOB, 0, 0, 0);
        };
        auto issue8 = [&](int s) {
            const int so = __builtin_amdgcn_readlane(ord, s);
            if (pfw) {
                if (s == q1 + 5) prefetch((hb * NP1 + 10) * BK + l12, true);
                if (s == q1 + 13) {
                    prefetch((hb * NP1 + 22) * BK + l12, l12 < (NP1 - 22) * BK);
                    prefetch(2 * NP1 * BK + l24, l24 < NP2 * BK);
                }
            }
            issue4(s, so, 0);
            issue4(s, so, 1);
        };
        issue8(1);
        VMW(8);
        const int nst = (lw == 3) ? NP1 : NS;
        BARRIER();
        issue8(2);
        issue8(3);
        VMW(16);
        for (int s = 1; s < nst - 3; ++s) {
            BARRIER();
            issue8(s + 3);
            VMW(16);
        }
        BARRIER();
        VMW(8);
        BARRIER();
        VMW(0);
        if (lw != 3) {
            BARRIER();
        } else {
            const unsigned call = RFL(c0) + 1;
            fl[F_CALL] = call;
            fl[F_AMAX + 1] = 0;
            fl[F_BAD] = 0;
            asm volatile("" : "+v"(pw));
            code_prefetch();
            WAIT_LGKM0();
            for (int s = NP1 - 1; s < NS; ++s) BARRIER();
        }
    } else {
        const int gq = lane >> 4, qq = (lane & 15) >> 2, pp = lane & 3;
        LAS char* bx[2][2][2];
        LAS char* bw[2][4];
#pragma unroll
        for (int hi = 0; hi < 2; ++hi) {
#pragma unroll
            for (int ks = 0; ks < 2; ++ks) {
                const int xo = hi * 65536 + (lane & 15) * 128 + (((4 * ks + gq) ^ ((lane >> 1) & 7)) << 4);
                bx[hi][ks][0] = shm + xo + wr * 12288;
                bx[hi][ks][1] = shm + xo + 4096 + wr * 4096;
                asm volatile("" : "+v"(bx[hi][ks][0]), "+v"(bx[hi][ks][1]));
            }
#pragma unroll
            for (int j = 0; j < 4; ++j) {
                bw[hi][j] = shm + hi * 65536 + W_OFF + wc * 8192 + 128 * (8 * gq + qq) + ((j ^ ((qq >> 1) | ((gq & 1) << 1))) << 5) + 8 * pp;
                asm volatile("" : "+v"(bw[hi][j]));
            }
        }
        f32x4 acc[4][4];
#pragma unroll
        for (int k = 0; k < 16; ++k) acc[k >> 2][k & 3] = (f32x4){0.f, 0.f, 0.f, 0.f};
        struct Frag { bf16x8 x[4], w[4]; };
        auto rd = [&](Frag& f, int sl, int ks, int ni) {
#pragma unroll
            for (int j = 0; j < 4; ++j) {
                if (j < ni) f.x[j] = *(LAS bf16x8*)(bx[sl >> 1][ks][j >> 1] + (sl & 1) * STAGE_B + (j & 1) * 2048);
                LAS char* wb = bw[sl >> 1][j] + (sl & 1) * STAGE_B + ks * 4096;
                const bf16x4 lo = __builtin_amdgcn_ds_read_tr16_b64_v4bf16((LAS bf16x4*)wb);
                const bf16x4 hi = __builtin_amdgcn_ds_read_tr16_b64_v4bf16((LAS bf16x4*)(wb + 512));
                f.w[j] = __builtin_shufflevector(lo, hi, 0, 1, 2, 3, 4, 5, 6, 7);
            }
        };
        auto mm = [&](const Frag& f, int n0, int n1) {
#pragma unroll
            for (int n = n0; n < n1; ++n)
                acc[n >> 2][n & 3] = __builtin_amdgcn_mfma_f32_16x16x32_bf16(f.w[n & 3], f.x[n >> 2], acc[n >> 2][n & 3], 0, 0, 0);
        };
#define SGB(m, n) __builtin_amdgcn_sched_group_barrier(m, n, 0)
#define SCHED_FENCE() __builtin_amdgcn_sched_barrier(0)
#define GROUP1_END() do { _Pragma("unroll") for (int _i = 0; _i < 12; ++_i) { SGB(0x8, 1); SGB(0x100, 1); } SGB(0x8, 4); SCHED_FENCE(); } while (0)
#define GROUP2_END() do { _Pragma("unroll") for (int _i = 0; _i < 5; ++_i) { SGB(0x8, 1); SGB(0x100, 2); } SGB(0x8, 3); SCHED_FENCE(); } while (0)
        Frag fa, fb;
        auto stage_barrier = [&](int npre) {
            __builtin_amdgcn_s_waitcnt(0xC07F);
            SCHED_FENCE();
            if (npre) mm(fb, 0, npre);
            SCHED_FENCE();
            BARRIER();
            SCHED_FENCE();
        };
        WAIT_VM(0);
        stage_barrier(0);
        rd(fa, 0, 0, 4);
        SCHED_FENCE();
        rd(fb, 0, 1, 4); mm(fa, 0, 16); GROUP1_END();
        auto stage1 = [&](int sl) {
            stage_barrier(PRE_MM);
            rd(fa, sl, 0, 4); mm(fb, PRE_MM, 16); GROUP1_END();
            rd(fb, sl, 1, 4); mm(fa, 0, 16); GROUP1_END();
        };
        for (int s = 1; s + 3 < NP1; s += 4) { stage1(1); stage1(2); stage1(3); stage1(0); }
#pragma unroll
        for (int s = NP1 - (NP1 - 1) % 4; s < NP1; ++s) stage1(s & 3);
#pragma unroll
        for (int k = 0; k < 4; ++k) bx[k >> 1][k & 1][0] -= wr * 8192;
        stage_barrier(PRE_MM);
        rd(fa, NP1 & 3, 0, 2);
        mm(fb, PRE_MM, 16);
        SCHED_FENCE();
        const bool giver = (wr != hb);
        const unsigned call = RFL(fl[F_CALL]), xtag = (call % 65535u + 1u) << 16;
        u32x4 pk[10];
        if (giver) {
#pragma unroll
            for (int g = 0; g < 10; ++g) {
                auto val = [&](int i) { const int v = (g % 5) * 7 + i; return v < 32 ? acc[(g / 5) * 2 + (v >> 4)][(v >> 2) & 3][v & 3] : 0.f; };
                pk[g] = (u32x4){pack2(val(0), val(1)), pack2(val(2), val(3)), pack2(val(4), val(5)), (pack2(val(6), 0.f) & 0xffffu) | xtag};
            }
#pragma unroll
            for (int k = 0; k < 8; ++k) acc[k >> 2][k & 3] = (f32x4){0.f, 0.f, 0.f, 0.f};
        }
        auto rdump = __builtin_amdgcn_make_buffer_rsrc((void*)(p.myws + WS_XCH + (size_t)b * 32768), 0, 32768, 0x00020000);
        auto rp = __builtin_amdgcn_make_buffer_rsrc((void*)(p.myws + WS_XCH + (size_t)(b ^ 128) * 32768), 0, 32768, 0x00020000);
        SCHED_FENCE();
        u32x4 r2[5];
        auto fetch = [&]() {
#pragma unroll
            for (int g = 0; g < 5; ++g) r2[g] = __builtin_amdgcn_raw_buffer_load_b128(rp, lane * 16, ((wc * 2 + wr) * 5 + g) * 1024, AUX_SYS);
        };
        auto fresh = [&]() { return __all((((r2[0][3] ^ xtag) | (r2[1][3] ^ xtag) | (r2[2][3] ^ xtag) | (r2[3][3] ^ xtag) | (r2[4][3] ^ xtag)) >> 16) == 0u); };
        auto add_r2 = [&]() {
#pragma unroll
            for (int v = 0; v < 32; ++v) {
                const unsigned d = r2[v / 7][(v % 7) >> 1];
                acc[v >> 4][(v >> 2) & 3][v & 3] += __uint_as_float(((v % 7) & 1) ? (d & 0xffff0000u) : (d << 16));
            }
        };
#pragma unroll
        for (int u = 0; u < NP2; ++u) {
            if (u > 0) { stage_barrier(0); rd(fa, (NP1 + u) & 3, 0, 2); mm(fb, 0, 8); GROUP2_END(); }
            if (u == NP2 - 1) {
                if (!fresh()) {
                    const u64 t0 = __builtin_amdgcn_s_memrealtime();
                    unsigned polls = 0;
                    for (;;) {
                        asm volatile("" ::: "memory");
                        fetch();
                        if (fresh()) break;
                        if (give_up(tmo, call, t0, polls)) { fl[F_BAD] = 1; r2[0][0] = 0x7f807f80u; break; }
                    }
                }
                add_r2();
                SCHED_FENCE();
            }
            rd(fb, (NP1 + u) & 3, 1, 2); mm(fa, 0, 8); GROUP2_END();
            if (u * GPS < 10) {
                if (giver) {
#pragma unroll
                    for (int g = u * GPS; g < (u + 1) * GPS; ++g)
                        __builtin_amdgcn_raw_buffer_store_b128(pk[g], rdump, lane * 16, ((wc * 2 + ((g / 5) ^ wr)) * 5 + g % 5) * 1024, AUX_SYS);
                    asm volatile("s_nop 1" ::: "memory");
                } else {
#pragma unroll
                    for (int k = u * (GPS * 8 / 10); k < (u + 1) * (GPS * 8 / 10); ++k)
                        *(LAS f32x4*)(shm + KB_OFF + wc * 8192 + k * 1024 + lane * 16) = acc[2 + (k >> 2)][k & 3];
                }
                SCHED_FENCE();
            }
            if (u == 3 && giver) {
#pragma unroll
                for (int k = 0; k < 8; ++k) acc[k >> 2][k & 3] += *(LAS f32x4*)(shm + KB_OFF + wc * 8192 + k * 1024 + lane * 16);
                SCHED_FENCE();
            }
            if (u == FETCH_SLOT) { fetch(); SCHED_FENCE(); }
        }
        mm(fb, 0, 8);
        float mx = 0.f;
#pragma unroll
        for (int k = 0; k < 8; ++k) {
            fin[k] = acc[k >> 2][k & 3];
            mx = fmaxf(fmaxf(mx, fin[k][0]), fin[k][1]);
            mx = fmaxf(fmaxf(mx, fin[k][2]), fin[k][3]);
        }
        fl[F_WMAX + wid] = wave_umax(__float_as_uint(mx) & 0x7fffffffu);
        WAIT_LGKM0();
    }
    BARRIER();
    const unsigned call = RFL(fl[F_CALL]);
    if (wid < 4) {
#pragma unroll
        for (int k = 0; k < 8; ++k) {
#pragma unroll
            for (int e = 0; e < 4; ++e) fin[k][e] = fmaxf(fin[k][e], 0.f);
            *(LAS f32x4*)(shm + r3_off(wc, wr * 8 + k) + ((lane ^ ((k & 3) << 2)) << 4)) = fin[k];
        }
        WAIT_LGKM0();
    } else if (wid == 7) {
        const u32x4 m4 = *(volatile LAS u32x4*)(shm + FL_OFF + F_WMAX * 4);
        const u64 gran = ((u64)call << 32) | umax(umax(m4[0], m4[1]), umax(m4[2], m4[3]));
        if (lane == 0) __hip_atomic_store((GAS u64*)(myws + WS_LMAX) + b, gran, RLX_AGENT);
    }
    BARRIER();
    const bool remote = grp != rank;
    const unsigned tagbits = spread4(call % 15u + 1u) << 7;
    const unsigned tag16 = call % 65535u + 1u;
    LAS char* cp = shm + 2 * STAGE_B + wid * 8192;
    auto ro = __builtin_amdgcn_make_buffer_rsrc((void*)out_ptr, 0, NR * ML * NL, 0x00020000);
    const int out0 = (grp * ML + tm * 128 + hb * 64 + (wid & 3) * 16 + (lane >> 3)) * NL + tn * 128 + (lane & 7) * 16;
    auto publish_scale = [&](unsigned am, bool bad) {
        const float amax = __uint_as_float(am);
        float inv = __builtin_amdgcn_rcpf(amax * (1.0f / 448.0f));
        if (!(amax > 1e-30f && amax < 1e30f)) inv = amax > 0.f ? 1.0f / (amax / 448.0f) : 0.f;
        *(volatile LAS u64*)(shm + FL_OFF + F_AMAX * 4) = ((u64)(bad ? 2u : 1u) << 32) | __float_as_uint(inv);
    };
    if (wid == 7) {
        const u64 t0 = __builtin_amdgcn_s_memrealtime();
        unsigned polls = 0;
        const size_t goff = WS_GMAX + (size_t)(call & 1) * (NREP * 128);
        if (b == rank) {
            GAS u64* lmax = (GAS u64*)(myws + WS_LMAX);
            unsigned gm;
            bool bad = false;
            for (;;) {
                bool ok = true;
                gm = 0;
#pragma unroll
                for (int k = 0; k < 4; ++k) {
                    const u64 v = __hip_atomic_load(lmax + k * 64 + lane, RLX_AGENT);
                    ok &= (unsigned)(v >> 32) >= call;
                    gm = umax(gm, (unsigned)v);
                }
                if (__all(ok)) break;
                if (give_up(tmo, call, t0, polls)) { bad = true; break; }
            }
            gm = wave_umax(gm);
            if (!bad) {
#pragma unroll
                for (int k = 0; k < NREP / 8; ++k)
                    __hip_atomic_store((GAS u64*)(pw + goff + (k * 8 + (lane & 7)) * 128) + rank, ((u64)call << 32) | gm, RLX_SYS);
            }
        }
        GAS u64* gp = (GAS u64*)(myws + goff + (b & (NREP - 1)) * 128) + (lane & 7);
        unsigned am;
        bool bad = fl[F_BAD] != 0u;
        for (;;) {
            const u64 v = __hip_atomic_load(gp, RLX_SYS);
            am = (unsigned)v;
            if (__all((unsigned)(v >> 32) >= call)) break;
            if (give_up(tmo, call, t0, polls)) { bad = true; break; }
        }
        publish_scale(wave_umax(am), bad);
    }
    auto quant = [&](const f32x4* v, float inv) {
        u32x4 o;
#pragma unroll
        for (int i = 0; i < 4; ++i) {
            const f32x4 t = v[i] * inv;
            const int w = __builtin_amdgcn_cvt_pk_fp8_f32(t[0], t[1], 0, false);
            o[i] = (unsigned)__builtin_amdgcn_cvt_pk_fp8_f32(t[2], t[3], w, true);
        }
        return o;
    };
    auto wait_scale = [&]() {
        u64 v;
        do v = *(volatile LAS u64*)(shm + FL_OFF + F_AMAX * 4); while (RFL((unsigned)(v >> 32)) == 0u);
        return v;
    };
    if (wid < 4) {
        f32x4 y[2][4];
#pragma unroll
        for (int h = 0; h < 2; ++h)
#pragma unroll
            for (int i = 0; i < 4; ++i)
                y[h][i] = *(LAS f32x4*)(shm + r3_off((lane >> 2) & 1, wid * 4 + (lane & 3)) +
                                        (((h * 8 + (lane >> 3) + 16 * i) ^ ((lane & 3) << 2)) << 4));
        int v0 = out0, v1 = out0 + 8 * NL;
        unsigned andb = 0xffffffffu, orb = 0u;
        auto rsnd = ro;
        if (remote) {
            unsigned mask[2] = {0u, 0u};
#pragma unroll
            for (int e = 0; e < 32; ++e) mask[e >> 4] |= (y[e >> 4][(e >> 2) & 3][e & 3] > 0.f ? 1u : 0u) << (e & 15);
            const int slot = idx * 8 + rank;
            GAS unsigned* mo = (GAS unsigned*)((GAS unsigned char*)peer_ws + WS_MASK + ((size_t)(call & 1) * GRID + slot) * 2048) + threadIdx.x;
            __hip_atomic_store(mo, mask[0] | (tag16 << 16), RLX_SYS);
            __hip_atomic_store(mo + 256, mask[1] | (tag16 << 16), RLX_SYS);
            const unsigned c = __builtin_popcount(mask[0]) | (__builtin_popcount(mask[1]) << 16), incl = wave_scan(c);
            const unsigned tot = __builtin_amdgcn_readlane(incl, 63), n = (tot & 0xffffu) + (tot >> 16);
#pragma unroll
            for (int h = 0; h < 2; ++h) {
                unsigned pos = (h ? (tot & 0xffffu) + ((incl - c) >> 16) : ((incl - c) & 0xffffu)) * 4;
#pragma unroll
                for (int e = 0; e < 16; ++e)
                    if (y[h][e >> 2][e & 3] > 0.f) { *(LAS float*)(cp + pos) = y[h][e >> 2][e & 3]; pos += 4; }
            }
            WAIT_LGKM0();
#pragma unroll
            for (int h = 0; h < 2; ++h)
#pragma unroll
                for (int i = 0; i < 4; ++i) y[h][i] = *(LAS f32x4*)(cp + h * 4096 + lane * 64 + i * 16);
            rsnd = __builtin_amdgcn_make_buffer_rsrc((void*)(peer_ws + WS_PAY + (size_t)slot * 8192), 0, 8192, 0x00020000);
            v0 = (unsigned)lane * 16 < n ? wid * 2048 + lane * 16 : OOB;
            v1 = (unsigned)lane * 16 + 1024 < n ? wid * 2048 + 1024 + lane * 16 : OOB;
            andb = 0x7f7f7f7fu;
            orb = tagbits;
        }
        asm volatile("" : "+v"(v0), "+v"(v1), "+s"(rsnd), "+s"(andb), "+s"(orb), "+v"(y[0][0]), "+v"(y[0][1]), "+v"(y[0][2]), "+v"(y[0][3]),
                          "+v"(y[1][0]), "+v"(y[1][1]), "+v"(y[1][2]), "+v"(y[1][3]));
        const u64 sv = wait_scale();
        const float inv = __uint_as_float((unsigned)sv);
        if ((unsigned)(sv >> 32) != 1u) orb |= 0x7f7f7f7fu;
        u32x4 o = quant(y[0], inv);
        __builtin_amdgcn_raw_buffer_store_b128((o & andb) | orb, rsnd, v0, 0, AUX_SYS);
        o = quant(y[1], inv);
        __builtin_amdgcn_raw_buffer_store_b128((o & andb) | orb, rsnd, v1, 0, AUX_SYS);
        asm volatile(".Lga_end_a:");
    } else if (remote) {
        const u64 t0 = __builtin_amdgcn_s_memrealtime();
        unsigned polls = 0;
        bool fail = false;
        GAS unsigned* mi = (GAS unsigned*)(myws + WS_MASK + ((size_t)(call & 1) * GRID + b) * 2048) + (threadIdx.x - 256);
        unsigned mw[2];
        for (;;) {
            mw[0] = __hip_atomic_load(mi, RLX_SYS);
            mw[1] = __hip_atomic_load(mi + 256, RLX_SYS);
            if (__all((mw[0] >> 16) == tag16 && (mw[1] >> 16) == tag16)) break;
            if (give_up(tmo, call, t0, polls)) { fail = true; break; }
        }
        const unsigned c = __builtin_popcount(mw[0] & 0xffffu) | (__builtin_popcount(mw[1] & 0xffffu) << 16), incl = wave_scan(c);
        const unsigned tot = __builtin_amdgcn_readlane(incl, 63), n = (tot & 0xffffu) + (tot >> 16);
        unsigned adr[2][4], sel[2][4];
        int voff[2];
#pragma unroll
        for (int h = 0; h < 2; ++h) {
            const unsigned mask = mw[h] & 0xffffu, p0 = h ? (tot & 0xffffu) + ((incl - c) >> 16) : ((incl - c) & 0xffffu);
            voff[h] = (unsigned)lane * 16 + h * 1024 < n ? (wid - 4) * 2048 + h * 1024 + lane * 16 : OOB;
#pragma unroll
            for (int j = 0; j < 4; ++j) {
                adr[h][j] = p0 + __builtin_popcount(mask & ((1u << (4 * j)) - 1u));
                const unsigned x = spread4((mask >> (4 * j)) & 15u);
                sel[h][j] = ((x * 0x01010100u) & (x * 0xffu)) | ((x ^ 0x01010101u) * 0x0cu);
                asm volatile("" : "+v"(adr[h][j]), "+v"(sel[h][j]));
            }
        }
        auto rrcv = __builtin_amdgcn_make_buffer_rsrc((void*)(p.myws + WS_PAY + (size_t)b * 8192), 0, 8192, 0x00020000);
        u32x4 v[2];
        v[0] = v[1] = (u32x4){0u, 0u, 0u, 0u};
        wait_scale();
        while (!fail) {
            asm volatile("" ::: "memory");
            bool ok = true;
#pragma unroll
            for (int h = 0; h < 2; ++h) {
                v[h] = __builtin_amdgcn_raw_buffer_load_b128(rrcv, voff[h], 0, AUX_SYS);
                const unsigned d = ((v[h][0] ^ tagbits) | (v[h][1] ^ tagbits) | (v[h][2] ^ tagbits) | (v[h][3] ^ tagbits)) & 0x80808080u;
                ok &= voff[h] == OOB || d == 0u;
            }
            if (__all(ok)) break;
            fail = give_up(tmo, call, t0, polls);
        }
#pragma unroll
        for (int h = 0; h < 2; ++h) *(LAS u32x4*)(cp + h * 1024 + lane * 16) = v[h];
        WAIT_LGKM0();
#pragma unroll
        for (int h = 0; h < 2; ++h) {
            u32x4 o;
#pragma unroll
            for (int j = 0; j < 4; ++j) {
                const unsigned a4 = adr[h][j] & ~3u;
                const unsigned lo = *(LAS unsigned*)(cp + a4), hi = *(LAS unsigned*)(cp + 4 + a4);
                const unsigned win = __builtin_amdgcn_alignbyte(hi, lo, adr[h][j] & 3u);
                o[j] = (__builtin_amdgcn_perm(0u, win, sel[h][j]) & 0x7f7f7f7fu) | (fail ? 0x7f7f7f7fu : 0u);
            }
            __builtin_amdgcn_raw_buffer_store_b128(o, ro, out0 + h * 8 * NL, 0, AUX_SYS);
        }
#pragma unroll
        for (int h = 0; h < 2; ++h)
            __builtin_amdgcn_raw_buffer_store_b128((u32x4){0u, 0u, 0u, 0u}, rrcv, voff[h], 0, AUX_SYS);
        asm volatile(".Lga_end_b:");
    }
    if (threadIdx.x == 0) __hip_atomic_store(cnt, call, RLX_AGENT);
    asm volatile(".Lga_end_c:");
#endif
}

}

extern "C" void kernel_launch(int rank, int nranks, ncclComm_t comm, hipStream_t stream,
                              void* const* d_in, const int* in_sizes, int n_in,
                              void* d_out, int out_size, const int* in_dtypes, int out_dtype,
                              void* const* d_out_all, void* const* d_ws_all, size_t ws_bytes) {
    (void)comm; (void)in_dtypes; (void)out_dtype; (void)d_out_all;
    if (nranks != ga::NR || (unsigned)rank >= (unsigned)ga::NR || n_in < 2 || in_sizes[0] != ga::ML * ga::KK ||
        in_sizes[1] != ga::KK * ga::NN || out_size != ga::NR * ga::ML * ga::NL || ws_bytes < ga::WS_END) {
        fprintf(stderr, "kernel_launch: unexpected problem shape\n");
        return;
    }
    HIP_CHECK(hipFuncSetAttribute((const void*)ga::gemm_a2a_kernel, hipFuncAttributeMaxDynamicSharedMemorySize, ga::LDS_BYTES));
    ga::Params p;
    p.x = d_in[0];
    p.w = d_in[1];
    for (int r = 0; r < ga::NR; ++r) p.ws[r] = (unsigned char*)d_ws_all[r];
    p.myws = p.ws[rank];
    p.myout = (unsigned char*)d_out;
    p.rank = rank;
    p.pad = 0;
    hipLaunchKernelGGL(ga::gemm_a2a_kernel, dim3(ga::GRID), dim3(ga::THREADS), ga::LDS_BYTES, stream, p);
    HIP_CHECK(hipGetLastError());
}
```
